```python
import numpy as np

kernel_name = "gpu_ag_gemm_m4096_k4096_n4096_f32_gelu"


ABSMAX_RTOL = 0.001

COMPUTE_DTYPE = "float32"
EPILOGUE = "gelu"
IN_QUANT = None
OUT_QUANT = None
IN_DTYPES = {"x": "float32", "w_mat": "float32"}
WORLD_SIZE = 8

def setup_inputs():

    x = np.random.randn(4096, 4096).astype(np.float32)
    w_mat = (np.random.randn(4096, 4096) / np.sqrt(4096)).astype(np.float32)
    return {"x": x, "w_mat": w_mat}

def _epilogue(y):
    c = 0.7978845608028654
    return (0.5 * y * (1.0 + np.tanh(c * (y + 0.044715 * y ** 3)))).astype(np.float32)

def reference(x, w_mat):
    y = (x @ w_mat).astype(np.float32)
    return _epilogue(y)
```

```cpp
#include <hip/hip_runtime.h>
#include <rccl/rccl.h>
#include <cstdio>
#include <cstdint>
#include <utility>

#define HIP_CHECK(expr) do { hipError_t _e = (expr); if (_e != hipSuccess) { \
    fprintf(stderr, "HIP error %s at %s:%d\n", hipGetErrorString(_e), __FILE__, __LINE__); \
    return; } } while (0)

typedef unsigned u32x4 __attribute__((ext_vector_type(4)));
typedef float f32x4 __attribute__((ext_vector_type(4)));
typedef _Float16 f16x8 __attribute__((ext_vector_type(8)));
#define LDS_AS __attribute__((address_space(3)))
#define CONST_AS __attribute__((address_space(4)))
#define GLOBAL_AS __attribute__((address_space(1)))

#define PUSH_NB 2
#define FLAG_STRIDE 128

constexpr int NRANK = 8;
constexpr int KDIM = 4096;
constexpr int SHARD = 512;
constexpr int GL = 8;
constexpr int NWG = 256;
constexpr int NTHREADS = 576;
constexpr unsigned PANEL = 1u << 20;
constexpr unsigned ROUND_B = 32768;
constexpr unsigned STEP_B = 16384;
constexpr int NSTEP = 64;

constexpr size_t OFF_CTR = 0;
constexpr size_t OFF_TMO = 32768;
constexpr size_t OFF_FLAG = 65536;
constexpr size_t OFF_OFLAG = 131072;
constexpr size_t OFF_DBG = 262144;
constexpr size_t OFF_DATA = 1u << 20;
constexpr size_t PARITY_B = 32u << 20;
constexpr int BSLOT0 = 17;
constexpr size_t WS_NEED = OFF_DATA + 2 * PARITY_B;
static_assert(GL * FLAG_STRIDE <= 1024, "flag block");

constexpr unsigned long long DEADLINE_TICKS = 40000000ull;

constexpr int LDS_RING = 147456;
constexpr int LDS_BYTES = LDS_RING + 64;

struct Params {
  const float* x;
  const float* w;
  char* ws[NRANK];
  float* out[NRANK];
  int rank;
  int pad;
};

template <int... I, class F>
__device__ __forceinline__ void static_for_impl(std::integer_sequence<int, I...>, F&& f) {
  (f(std::integral_constant<int, I>{}), ...);
}
template <int N, class F>
__device__ __forceinline__ void static_for(F&& f) {
  static_for_impl(std::make_integer_sequence<int, N>{}, f);
}

#define BARRIER() do { asm volatile("" ::: "memory"); __builtin_amdgcn_s_waitcnt(0xC07F); \
    asm volatile("s_barrier" ::: "memory"); } while (0)

#define STAMP(dbg, id) do { } while (0)

__device__ __forceinline__ unsigned pk16(float a, float b) {
  typedef _Float16 f16x2 __attribute__((ext_vector_type(2)));
  f16x2 h;
  h[0] = (_Float16)a;
  h[1] = (_Float16)b;
  return __builtin_bit_cast(unsigned, h);
}

__device__ __forceinline__ void flag_store(unsigned* p, unsigned v) {
  __hip_atomic_store((GLOBAL_AS unsigned*)p, v, __ATOMIC_RELAXED, __HIP_MEMORY_SCOPE_SYSTEM);
}
__device__ __forceinline__ unsigned flag_load(const unsigned* p) {
  return __hip_atomic_load((GLOBAL_AS unsigned*)p, __ATOMIC_RELAXED, __HIP_MEMORY_SCOPE_SYSTEM);
}

__device__ __forceinline__ void load_w(f32x4 (&b)[8], unsigned v0, unsigned v1, const char* s) {
  asm volatile("s_nop 4\n\t"
               "global_load_dwordx4 %0, %8, %10 offset:-4096\n\t"
               "global_load_dwordx4 %1, %8, %10 offset:-2048\n\t"
               "global_load_dwordx4 %2, %8, %10\n\t"
               "global_load_dwordx4 %3, %8, %10 offset:2048\n\t"
               "global_load_dwordx4 %4, %9, %10 offset:-4096\n\t"
               "global_load_dwordx4 %5, %9, %10 offset:-2048\n\t"
               "global_load_dwordx4 %6, %9, %10\n\t"
               "global_load_dwordx4 %7, %9, %10 offset:2048"
               : "=&v"(b[0]), "=&v"(b[1]), "=&v"(b[2]), "=&v"(b[3]), "=&v"(b[4]), "=&v"(b[5]), "=&v"(b[6]), "=&v"(b[7])
               : "v"(v0), "v"(v1), "s"(s) : "memory");
}
__device__ __forceinline__ void load_x(f32x4 (&b)[8], unsigned v0, const char* s) {
  asm volatile("s_nop 4\n\t"
               "global_load_dwordx4 %0, %8, %9\n\t"
               "global_load_dwordx4 %1, %8, %9 offset:64\n\t"
               "global_load_dwordx4 %2, %8, %9 offset:128\n\t"
               "global_load_dwordx4 %3, %8, %9 offset:192\n\t"
               "global_load_dwordx4 %4, %8, %9 offset:256\n\t"
               "global_load_dwordx4 %5, %8, %9 offset:320\n\t"
               "global_load_dwordx4 %6, %8, %9 offset:384\n\t"
               "global_load_dwordx4 %7, %8, %9 offset:448"
               : "=&v"(b[0]), "=&v"(b[1]), "=&v"(b[2]), "=&v"(b[3]), "=&v"(b[4]), "=&v"(b[5]), "=&v"(b[6]), "=&v"(b[7])
               : "v"(v0), "s"(s) : "memory");
}
template <int CNT>
__device__ __forceinline__ void wait_loads(f32x4 (&b)[8]) {
  asm volatile("s_waitcnt vmcnt(%8)"
               : "+v"(b[0]), "+v"(b[1]), "+v"(b[2]), "+v"(b[3]), "+v"(b[4]), "+v"(b[5]), "+v"(b[6]), "+v"(b[7])
               : "i"(CNT) : "memory");
}
template <int CNT>
__device__ __forceinline__ void wait_vm() {
  asm volatile("s_waitcnt vmcnt(%0)" :: "i"(CNT) : "memory");
}
__device__ __forceinline__ void store_w(const f32x4 (&b)[8], unsigned v0, char* d) {
  u32x4 o[4];
  for (int n = 0; n < 4; ++n)
    for (int e = 0; e < 4; ++e) o[n][e] = pk16(b[2 * e][n], b[2 * e + 1][n]);
  asm volatile("global_store_dwordx4 %4, %0, %5 sc0 sc1\n\t"
               "global_store_dwordx4 %4, %1, %5 offset:1024 sc0 sc1\n\t"
               "global_store_dwordx4 %4, %2, %5 offset:2048 sc0 sc1\n\t"
               "global_store_dwordx4 %4, %3, %5 offset:3072 sc0 sc1\n\ts_nop 1"
               :: "v"(o[0]), "v"(o[1]), "v"(o[2]), "v"(o[3]), "v"(v0), "s"(d) : "memory");
}
__device__ __forceinline__ void store_x(const f32x4 (&b)[8], unsigned v0, unsigned v1, char* d) {
  u32x4 o[4];
  for (int u = 0; u < 4; ++u) {
    o[u][0] = pk16(b[2 * u][0], b[2 * u][1]);
    o[u][1] = pk16(b[2 * u][2], b[2 * u][3]);
    o[u][2] = pk16(b[2 * u + 1][0], b[2 * u + 1][1]);
    o[u][3] = pk16(b[2 * u + 1][2], b[2 * u + 1][3]);
  }
  asm volatile("global_store_dwordx4 %4, %0, %6 sc0 sc1\n\t"
               "global_store_dwordx4 %4, %1, %6 offset:1024 sc0 sc1\n\t"
               "global_store_dwordx4 %5, %2, %6 sc0 sc1\n\t"
               "global_store_dwordx4 %5, %3, %6 offset:1024 sc0 sc1\n\ts_nop 1"
               :: "v"(o[0]), "v"(o[1]), "v"(o[2]), "v"(o[3]), "v"(v0), "v"(v1), "s"(d) : "memory");
}

template <int NW, int NX>
__device__ __forceinline__ void push_stream(const char* wsrc, const char* xsrc, char* wdst, char* xdst,
                                            unsigned* flag, unsigned call, int g, int wave, int lane,
                                            unsigned long long* dbg) {
  constexpr int N = NW + NX;
  constexpr int NB = PUSH_NB < N ? PUSH_NB : N;
  if (wave == 8) {
    for (int i = 0; i < N; ++i) {
      BARRIER();
      if (lane == 0) flag_store(flag, call * 256u + (unsigned)(i + 1));
      STAMP(dbg, 2 + i);
    }
    return;
  }
  const int q = lane >> 4, c = lane & 15;
  const unsigned xl = (unsigned)(wave * 16 + c) * (KDIM * 4) + q * 16;
  const unsigned wl0 = (unsigned)((wave >> 1) * 32 + 4 * q + 2) * (SHARD * 4) + (wave & 1) * 256 + c * 16;
  const unsigned wl1 = wl0 + 16 * (SHARD * 4);
  const unsigned xs0 = wave * 2048 + lane * 16, xs1 = xs0 + STEP_B;
  const unsigned ws0 = (wave >> 2) * STEP_B + (wave & 1) * 8192 + ((wave >> 1) & 1) * 4096 + lane * 16;
  f32x4 buf[NB][8];

  auto load = [&](auto ic) {
    constexpr int i = decltype(ic)::value;
    if constexpr (i < NW) {
      load_w(buf[i % NB], wl0, wl1, wsrc + (size_t)(g + i * GL) * (128 * SHARD * 4));
    } else {
      constexpr int xr = (i - NW) * GL;
      load_x(buf[i % NB], xl, xsrc + (size_t)(xr / 32) * (128 * KDIM * 4) + (size_t)(g + xr % 32) * 512);
    }
  };

  static_for<NB>([&](auto ic) { load(ic); });
  static_for<N>([&](auto ic) {
    constexpr int i = decltype(ic)::value;
    constexpr int ns = i < NB - 1 ? i : NB - 1;
    constexpr int nl = (i + NB - 1 < N - 1 ? i + NB - 1 : N - 1) - i;
    wait_loads<ns * 4 + nl * 8>(buf[i % NB]);
    if constexpr (i >= NB) BARRIER();
    if constexpr (i < NW) store_w(buf[i % NB], ws0, wdst + (size_t)(g + i * GL) * ROUND_B);
    else store_x(buf[i % NB], xs0, xs1, xdst + (size_t)(g + (i - NW) * GL) * ROUND_B);
    if constexpr (i + NB < N) load(std::integral_constant<int, i + NB>{});
  });
  static_for<NB>([&](auto jc) {
    constexpr int j = decltype(jc)::value;
    wait_vm<(NB - 1 - j) * 4>();
    BARRIER();
  });
}

__device__ __forceinline__ float gelu(float y) {
  const float k1 = -2.0f * 0.7978845608028654f * 1.4426950408889634f;
  const float k3 = k1 * 0.044715f;
  float e = __builtin_amdgcn_exp2f(y * (k1 + k3 * y * y));
  return y * __builtin_amdgcn_rcpf(1.0f + e);
}

template <int MT>
__device__ __forceinline__ void gemm_tile(LDS_AS char* smem, const char* asrc, const char* bsrc,
                                          const unsigned* aflag, int abase, const unsigned* bflag, int bbase,
                                          char* out, unsigned* oflag, unsigned* tmo, unsigned call,
                                          int wave, int lane, unsigned long long* dbg) {
  constexpr int SLOT = MT * 8192 + 8192;
  constexpr int NS = MT == 2 ? 6 : 8;
  constexpr int D = NS - 1;
  constexpr int NLD = 2 * MT + 2;
  static_assert(NS * SLOT <= LDS_RING, "ring");
  LDS_AS unsigned* bad = (LDS_AS unsigned*)(smem + LDS_RING);

  if (wave == 8) {
    const int g = lane & 7;
    const bool act = lane < 16;
    const unsigned* fp = (const unsigned*)((const char*)(lane < 8 ? aflag : bflag) + g * FLAG_STRIDE);
    const int base = lane < 8 ? abase : bbase;
    int first = act ? -1 : 1 << 20;
    bool dead = false;
    STAMP(dbg, 0);
    for (int t = 0; t < NSTEP; ++t) {
      const int need = t >> 1;
      if (!dead && !__all(first > need)) {
        const unsigned long long t0 = __builtin_amdgcn_s_memrealtime();
        for (;;) {
          if (first <= need) {
            const int d = (int)(flag_load(fp) - call * 256u);
            first = (d > 0 ? d : 0) * GL + g - base;
          }
          if (__all(first > need)) break;
          const unsigned tv = __hip_atomic_load((GLOBAL_AS unsigned*)tmo, __ATOMIC_RELAXED, __HIP_MEMORY_SCOPE_AGENT);
          if (__builtin_amdgcn_readfirstlane(tv) == call || __builtin_amdgcn_s_memrealtime() - t0 > DEADLINE_TICKS) {
            if (lane == 0) __hip_atomic_store((GLOBAL_AS unsigned*)tmo, call, __ATOMIC_RELAXED, __HIP_MEMORY_SCOPE_AGENT);
            dead = true;
            break;
          }
          __builtin_amdgcn_s_sleep(4);
        }
      }
      BARRIER();
    }
    {
      const unsigned tv = __hip_atomic_load((GLOBAL_AS unsigned*)tmo, __ATOMIC_RELAXED, __HIP_MEMORY_SCOPE_AGENT);
      if (lane == 0) *bad = (dead || tv == call) ? 0x7fc00000u : 0u;
    }
    for (int t = 0; t < D; ++t) BARRIER();
    STAMP(dbg, 4);
    BARRIER();
    if (oflag != nullptr && lane == 0) flag_store(oflag, call);
    STAMP(dbg, 5);
  } else if (wave >= 4) {
    const int j = wave - 4;
    const unsigned voff = lane * 16;
    const char* a = asrc + j * (MT * 2048);
    const char* b = bsrc + j * 2048;
    const unsigned lbase = (unsigned)(uintptr_t)smem;
    unsigned slot = 0;
    for (int t = 0; t < NSTEP; ++t) {
      BARRIER();
      const unsigned la = lbase + slot + j * (MT * 2048);
      const unsigned lb = lbase + slot + MT * 8192 + j * 2048;
      unsigned keep;
      if constexpr (MT == 2) {
        asm volatile("s_mov_b32 %0, m0\n\ts_mov_b32 m0, %4\n\ts_nop 0\n\t"
                     "global_load_lds_dwordx4 %1, %2 sc0 sc1\n\t"
                     "global_load_lds_dwordx4 %1, %2 offset:1024 sc0 sc1\n\t"
                     "global_load_lds_dwordx4 %1, %2 offset:2048 sc0 sc1\n\t"
                     "global_load_lds_dwordx4 %1, %2 offset:3072 sc0 sc1\n\t"
                     "s_mov_b32 m0, %5\n\ts_nop 0\n\t"
                     "global_load_lds_dwordx4 %1, %3 sc0 sc1\n\t"
                     "global_load_lds_dwordx4 %1, %3 offset:1024 sc0 sc1\n\t"
                     "s_mov_b32 m0, %0\n\ts_waitcnt vmcnt(%6)"
                     : "=&s"(keep) : "v"(voff), "s"(a), "s"(b), "s"(la), "s"(lb), "i"((D - 1) * NLD) : "memory");
      } else {
        asm volatile("s_mov_b32 %0, m0\n\ts_mov_b32 m0, %4\n\ts_nop 0\n\t"
                     "global_load_lds_dwordx4 %1, %2 sc0 sc1\n\t"
                     "global_load_lds_dwordx4 %1, %2 offset:1024 sc0 sc1\n\t"
                     "s_mov_b32 m0, %5\n\ts_nop 0\n\t"
                     "global_load_lds_dwordx4 %1, %3 sc0 sc1\n\t"
                     "global_load_lds_dwordx4 %1, %3 offset:1024 sc0 sc1\n\t"
                     "s_mov_b32 m0, %0\n\ts_waitcnt vmcnt(%6)"
                     : "=&s"(keep) : "v"(voff), "s"(a), "s"(b), "s"(la), "s"(lb), "i"((D - 1) * NLD) : "memory");
      }
      a += STEP_B;
      b += STEP_B;
      slot += SLOT;
      if (slot == NS * SLOT) slot = 0;
    }
    for (int t = 0; t < D; ++t) {
      BARRIER();
      asm volatile("s_waitcnt vmcnt(0)" ::: "memory");
    }
    BARRIER();
  } else {
    f32x4 acc[MT][4];
    for (int m = 0; m < MT; ++m)
      for (int n = 0; n < 4; ++n) acc[m][n] = f32x4{0.f, 0.f, 0.f, 0.f};
    f16x8 fa[2][MT][2], fb[2][4][2];
    LDS_AS char* pa = smem + wave * (MT * 2048) + lane * 16;
    LDS_AS char* pb = smem + MT * 8192 + lane * 16;
    unsigned slot = 0;
    auto rd = [&](auto sc) {
      constexpr int s = decltype(sc)::value;
      for (int u = 0; u < 2; ++u) {
        for (int m = 0; m < MT; ++m) fa[s][m][u] = *(LDS_AS f16x8*)(pa + slot + m * 2048 + u * 1024);
        for (int n = 0; n < 4; ++n) fb[s][n][u] = *(LDS_AS f16x8*)(pb + slot + u * 4096 + n * 1024);
      }
      slot += SLOT;
      if (slot == NS * SLOT) slot = 0;
    };
    auto mm = [&](auto sc) {
      constexpr int s = decltype(sc)::value;
      for (int u = 0; u < 2; ++u)
        for (int m = 0; m < MT; ++m)
          for (int n = 0; n < 4; ++n)
            acc[m][n] = __builtin_amdgcn_mfma_f32_16x16x32_f16(fa[s][m][u], fb[s][n][u], acc[m][n], 0, 0, 0);
    };
    using I0 = std::integral_constant<int, 0>;
    using I1 = std::integral_constant<int, 1>;
    for (int t = 0; t < D; ++t) BARRIER();
    BARRIER();
    rd(I0{});
    for (int s = 1; s < NSTEP - 1; s += 2) {
      BARRIER();
      rd(I1{});
      __builtin_amdgcn_sched_barrier(0);
      mm(I0{});
      __builtin_amdgcn_sched_barrier(0);
      BARRIER();
      rd(I0{});
      __builtin_amdgcn_sched_barrier(0);
      mm(I1{});
      __builtin_amdgcn_sched_barrier(0);
    }
    BARRIER();
    rd(I1{});
    const unsigned isbad = *bad;
    mm(I0{});
    mm(I1{});
    const unsigned vo = (unsigned)(wave * (16 * MT) + 4 * (lane >> 4) + 2) * (SHARD * 4) + (lane & 15) * 16;
    for (int m = 0; m < MT; ++m) {
      u32x4 o[4];
      for (int jj = 0; jj < 4; ++jj)
        for (int n = 0; n < 4; ++n)
          o[jj][n] = isbad | __builtin_bit_cast(unsigned, gelu(acc[m][n][jj]));
      const unsigned vm = vo + m * 16 * (SHARD * 4);
      asm volatile("global_store_dwordx4 %4, %0, %5 offset:-4096 sc0 sc1\n\t"
                   "global_store_dwordx4 %4, %1, %5 offset:-2048 sc0 sc1\n\t"
                   "global_store_dwordx4 %4, %2, %5 sc0 sc1\n\t"
                   "global_store_dwordx4 %4, %3, %5 offset:2048 sc0 sc1\n\ts_nop 1"
                   :: "v"(o[0]), "v"(o[1]), "v"(o[2]), "v"(o[3]), "v"(vm), "s"(out) : "memory");
    }
    asm volatile("s_waitcnt vmcnt(0)" ::: "memory");
    BARRIER();
  }
}

template <int MT>
__device__ __forceinline__ void run_tile(const Params& P, LDS_AS char* smem, int aslot, int ct, int mh,
                                         unsigned call, int wave, int lane, unsigned long long* dbg) {
  const int grp = P.rank >> 2, pos = P.rank & 3;
  char* ws = P.ws[P.rank];
  char* data = ws + OFF_DATA + (size_t)(call & 1) * PARITY_B;
  const int bslot = ct >> 1, ch = ct & 1;
  int astream, abase, row0;
  if (aslot < 3) { astream = aslot; abase = 0; row0 = P.rank * SHARD + aslot * 128; }
  else if (aslot < 12) {
    const int i = aslot / 3, b = aslot - 3 * i;
    astream = 7 + i; abase = b * 32; row0 = (grp * 4 + ((pos + i) & 3)) * SHARD + b * 128;
  } else if (aslot < 16) {
    astream = aslot; abase = 32; row0 = ((grp ^ 1) * 4 + ((pos + aslot - 12) & 3)) * SHARD + 384;
  } else { astream = 3; abase = 0; row0 = P.rank * SHARD + 384; }
  int bstream, drank, col0;
  unsigned* oflag = nullptr;
  if (bslot < 4) { bstream = 4 + bslot; drank = P.rank; col0 = bslot * 128 + ch * 64; }
  else {
    const int c = bslot - 4, cr = (4 - c) & 3;
    bstream = 12 + c; drank = (grp ^ 1) * 4 + ((pos + c) & 3); col0 = cr * 128 + ch * 64;
    oflag = (unsigned*)(P.ws[drank & 7] + OFF_OFLAG + (size_t)(cr * 64 + (aslot * 2 + ch) * 2 + mh) * 128);
  }
  row0 += mh * 64;
  const char* asrc = data + (size_t)aslot * PANEL + mh * 8192;
  const char* bsrc = data + (size_t)(BSLOT0 + bslot) * PANEL + ch * 8192;
  char* out = (char*)P.out[drank & 7] + (size_t)row0 * (SHARD * 4) + col0 * 4;
  gemm_tile<MT>(smem, asrc, bsrc, (const unsigned*)(ws + OFF_FLAG + astream * 1024), abase,
                (const unsigned*)(ws + OFF_FLAG + bstream * 1024), 0, out, oflag,
                (unsigned*)(ws + OFF_TMO), call, wave, lane, dbg);
}

__global__ void __launch_bounds__(NTHREADS, 1) ag_gemm_kernel(Params P) {
  __shared__ __attribute__((aligned(16))) char smem_raw[LDS_BYTES];
  LDS_AS char* smem = (LDS_AS char*)smem_raw;
  const int lane = threadIdx.x & 63;
  const int wave = __builtin_amdgcn_readfirstlane(threadIdx.x >> 6);
  const int bx = blockIdx.x, xcd = bx & 7, j = bx >> 3;
  const int grp = P.rank >> 2, pos = P.rank & 3;
  char* ws = P.ws[P.rank];
  unsigned* ctr = (unsigned*)(ws + OFF_CTR + (size_t)bx * 128);
  const unsigned call = *(const CONST_AS unsigned*)(uintptr_t)ctr + 1u;
  const size_t dpar = OFF_DATA + (size_t)(call & 1) * PARITY_B;
  unsigned long long* dbg = (unsigned long long*)(ws + OFF_DBG + (size_t)bx * 512);
  if (wave == 8) STAMP(dbg, 0);

  if (j < 3) {
    const int i = j + 1, ir = 4 - i;
    char* pw = P.ws[(grp * 4 + ((pos + i) & 3)) & 7];
    push_stream<0, 96 / GL>(nullptr, (const char*)P.x, nullptr, pw + dpar + (size_t)(ir * 3) * PANEL,
                            (unsigned*)(pw + OFF_FLAG + (7 + ir) * 1024 + xcd * FLAG_STRIDE), call, xcd, wave, lane, dbg);
  } else if (j < 7) {
    const int c = j - 3, cr = (4 - c) & 3;
    char* pw = P.ws[((grp ^ 1) * 4 + ((pos + c) & 3)) & 7];
    push_stream<32 / GL, 32 / GL>((const char*)P.w + c * 512, (const char*)P.x + (size_t)384 * KDIM * 4,
                                  pw + dpar + (size_t)(BSLOT0 + 4 + cr) * PANEL, pw + dpar + (size_t)(12 + cr) * PANEL,
                                  (unsigned*)(pw + OFF_FLAG + (12 + cr) * 1024 + xcd * FLAG_STRIDE), call, xcd, wave, lane, dbg);
    if (xcd == 0 && wave == 8) {
      const int aslot = lane >> 2, mh = lane & 1;
      const bool half = aslot == 5 || aslot == 8 || aslot == 11;
      bool done = aslot == 12 + cr || (mh == 1 && !half);
      const unsigned* fp = (const unsigned*)(ws + OFF_OFLAG + (size_t)(c * 64 + lane) * 128);
      unsigned* tmo = (unsigned*)(ws + OFF_TMO);
      const unsigned long long t0 = __builtin_amdgcn_s_memrealtime();
      for (;;) {
        if (!done) done = (int)(flag_load(fp) - call) >= 0;
        if (__all(done)) break;
        const unsigned tv = __hip_atomic_load((GLOBAL_AS unsigned*)tmo, __ATOMIC_RELAXED, __HIP_MEMORY_SCOPE_AGENT);
        if (__builtin_amdgcn_readfirstlane(tv) == call || __builtin_amdgcn_s_memrealtime() - t0 > DEADLINE_TICKS) {
          if (lane == 0) {
            __hip_atomic_store((GLOBAL_AS unsigned*)tmo, call, __ATOMIC_RELAXED, __HIP_MEMORY_SCOPE_AGENT);
            flag_store((unsigned*)(P.out[P.rank] + (size_t)P.rank * SHARD * SHARD), 0x7fc00000u);
          }
          break;
        }
        __builtin_amdgcn_s_sleep(4);
      }
      STAMP(dbg, 30);
    }
  } else {
    const int r = j - 7;
    int a0 = -1, c0 = 0, a1 = -1, c1 = 0, h2 = -1;
    if (r < 9) {
      const int pair = 3 * xcd + r / 3, b = r % 3;
      if (pair < 8) { a0 = b; c0 = 8 + pair; } else { a0 = 3 + 3 * b; c0 = pair - 8; }
      h2 = r;
    } else if (r < 22) {
      const int rr = r - 9;
      if (rr < 4) {
        push_stream<0, 32 / GL>(nullptr, (const char*)P.x + (size_t)rr * 128 * KDIM * 4, nullptr,
                                ws + dpar + (size_t)(rr < 3 ? rr : 16) * PANEL,
                                (unsigned*)(ws + OFF_FLAG + rr * 1024 + xcd * FLAG_STRIDE), call, xcd, wave, lane, dbg + 40);
      } else if (rr < 8) {
        push_stream<32 / GL, 0>((const char*)P.w + (rr - 4) * 512, nullptr, ws + dpar + (size_t)(BSLOT0 + rr - 4) * PANEL,
                                nullptr, (unsigned*)(ws + OFF_FLAG + rr * 1024 + xcd * FLAG_STRIDE), call, xcd, wave, lane, dbg + 40);
      }
      const int it = 13 * xcd + rr;
      if (it < 48) { c0 = it / 3; a0 = 4 + 3 * (it % 3); }
      else if (it < 72) {
        const int k = it - 48;
        c0 = 8 + k / 3;
        a0 = 12 + ((((c0 - 8) >> 1) + 1 + k % 3) & 3);
      } else { const int k = it - 72; c0 = k >> 2; a0 = 12 + (k & 3); }
    } else {
      const int e = r - 22;
      a0 = e; c0 = xcd;
      if (e == 0) { a1 = 16; c1 = xcd; }
      h2 = 9 + e;
    }
    for (int n = 0; n < 2; ++n) {
      const int a = n == 0 ? a0 : a1, c = n == 0 ? c0 : c1;
      if (a >= 0) run_tile<2>(P, smem, a, c, 0, call, wave, lane, dbg + 8 + 8 * n);
    }
    if (h2 >= 0) run_tile<1>(P, smem, 5 + 3 * ((h2 % 6) >> 1), 2 * xcd + h2 / 6, h2 & 1, call, wave, lane, dbg + 24);
  }
  if (wave == 8) {
    if (lane == 0) __hip_atomic_store((GLOBAL_AS unsigned*)ctr, call, __ATOMIC_RELAXED, __HIP_MEMORY_SCOPE_AGENT);
    STAMP(dbg, 1);
  }
}

extern "C" void kernel_launch(int rank, int nranks, ncclComm_t comm,
                              hipStream_t stream,
                              void* const* d_in, const int* in_sizes, int n_in,
                              void* d_out, int out_size,
                              const int* in_dtypes, int out_dtype,
                              void* const* d_out_all,
                              void* const* d_ws_all, size_t ws_bytes) {
  (void)comm; (void)in_dtypes; (void)out_dtype; (void)d_out;
  if (nranks != NRANK || (unsigned)rank >= (unsigned)NRANK || n_in < 2 || ws_bytes < WS_NEED ||
      in_sizes[0] != SHARD * KDIM || in_sizes[1] != KDIM * SHARD || out_size != NRANK * SHARD * SHARD) {
    fprintf(stderr, "unexpected problem shape\n");
    return;
  }
  Params P;
  P.x = (const float*)d_in[0];
  P.w = (const float*)d_in[1];
  for (int p = 0; p < NRANK; ++p) {
    P.ws[p] = (char*)d_ws_all[p];
    P.out[p] = (float*)d_out_all[p];
  }
  P.rank = rank;
  P.pad = 0;
  ag_gemm_kernel<<<NWG, NTHREADS, 0, stream>>>(P);
  HIP_CHECK(hipGetLastError());
}
```

```cpp
#include <hip/hip_runtime.h>
#include <rccl/rccl.h>
#include <cstdio>
#include <cstdint>
#include <utility>

#define HIP_CHECK(expr) do { hipError_t _e = (expr); if (_e != hipSuccess) { \
    fprintf(stderr, "HIP error %s at %s:%d\n", hipGetErrorString(_e), __FILE__, __LINE__); \
    return; } } while (0)

typedef unsigned u32x4 __attribute__((ext_vector_type(4)));
typedef float f32x4 __attribute__((ext_vector_type(4)));
typedef _Float16 f16x8 __attribute__((ext_vector_type(8)));
#define LDS_AS __attribute__((address_space(3)))
#define CONST_AS __attribute__((address_space(4)))
#define GLOBAL_AS __attribute__((address_space(1)))

#define U2 3
#define PMAP 0
#define SW 1
#define FLAG_STRIDE 128
#define POLL_SLEEP 4
#define CONV_DELAY 0

constexpr int NRANK = 8;
constexpr int KDIM = 4096;
constexpr int SHARD = 512;
constexpr int UX = 8 - U2;
constexpr int ROWX = 256 + 32 * U2;
constexpr int NWV = 16;
constexpr int PW = PMAP == 0 ? 2 : 4;
constexpr int JP = PMAP == 0 ? 7 : 4;
constexpr int NWG = 256;
constexpr int NTHREADS = 640;
constexpr unsigned MIB = 1u << 20;
constexpr int NSTEP = 64;
constexpr int NSTG = 2 * (32 + U2 + 3 * UX);
constexpr int NDIR = 2 * 3 * U2;

constexpr size_t OFF_CTR = 0;
constexpr size_t OFF_TMO = 32768;
constexpr size_t OFF_FLAG = 65536;
constexpr size_t OFF_OFLAG = 98304;
constexpr size_t OFF_SFLAG = 131072;
constexpr size_t OFF_DBG = 262144;
constexpr size_t OFF_DATA = MIB;
constexpr size_t PARITY_B = 40 * MIB;
constexpr size_t D_OWNX = 0, D_OWNW = 4 * MIB, D_INTRA = 8 * MIB, D_CROSS = 17 * MIB, D_STG = 29 * MIB;
constexpr size_t WS_NEED = OFF_DATA + 2 * PARITY_B;
static_assert(NWV * FLAG_STRIDE <= 2048 && NSTG <= 128 && 16 + NDIR <= 64, "flag blocks");

constexpr unsigned long long DEADLINE_TICKS = 40000000ull;

constexpr int LDS_RING = 147456;
constexpr int LDS_BYTES = LDS_RING + 64;

struct Params {
  const float* x;
  const float* w;
  char* self;
  char* ws[NRANK];
  float* out[NRANK];
  int rank;
  int pad;
};

template <int... I, class F>
__device__ __forceinline__ void static_for_impl(std::integer_sequence<int, I...>, F&& f) {
  (f(std::integral_constant<int, I>{}), ...);
}
template <int N, class F>
__device__ __forceinline__ void static_for(F&& f) {
  static_for_impl(std::make_integer_sequence<int, N>{}, f);
}

#define BARRIER() do { asm volatile("" ::: "memory"); __builtin_amdgcn_s_waitcnt(0xC07F); \
    asm volatile("s_barrier" ::: "memory"); } while (0)

#define STAMP(dbg, id) do { } while (0)

__device__ __forceinline__ unsigned pk16(float a, float b) {
  typedef _Float16 f16x2 __attribute__((ext_vector_type(2)));
  f16x2 h;
  h[0] = (_Float16)a;
  h[1] = (_Float16)b;
  return __builtin_bit_cast(unsigned, h);
}

__device__ __forceinline__ void flag_store(void* p, unsigned v) {
  __hip_atomic_store((GLOBAL_AS unsigned*)p, v, __ATOMIC_RELAXED, __HIP_MEMORY_SCOPE_SYSTEM);
}
__device__ __forceinline__ void poll_once(unsigned& f, unsigned& tv, const char* fp, const char* tmo) {
  asm volatile("global_load_dword %0, %2, off sc0 sc1\n\t"
               "global_load_dword %1, %3, %4 sc1\n\ts_waitcnt vmcnt(0)"
               : "=&v"(f), "=&v"(tv) : "v"(fp), "v"(0u), "s"(tmo) : "memory");
}
__device__ __forceinline__ bool expired(unsigned tv, char* tmo, unsigned call, unsigned long long t0, int lane) {
  if (__builtin_amdgcn_readfirstlane(tv) == call || __builtin_amdgcn_s_memrealtime() - t0 > DEADLINE_TICKS) {
    if (lane == 0) __hip_atomic_store((GLOBAL_AS unsigned*)tmo, call, __ATOMIC_RELAXED, __HIP_MEMORY_SCOPE_AGENT);
    return true;
  }
  return false;
}

__device__ __forceinline__ void load_unit(f32x4 (&b)[8], const unsigned (&v)[8], const char* s) {
  asm volatile("global_load_dwordx4 %0, %8, %16\n\t"
               "global_load_dwordx4 %1, %9, %16\n\t"
               "global_load_dwordx4 %2, %10, %16\n\t"
               "global_load_dwordx4 %3, %11, %16\n\t"
               "global_load_dwordx4 %4, %12, %16\n\t"
               "global_load_dwordx4 %5, %13, %16\n\t"
               "global_load_dwordx4 %6, %14, %16\n\t"
               "global_load_dwordx4 %7, %15, %16"
               : "=&v"(b[0]), "=&v"(b[1]), "=&v"(b[2]), "=&v"(b[3]), "=&v"(b[4]), "=&v"(b[5]), "=&v"(b[6]), "=&v"(b[7])
               : "v"(v[0]), "v"(v[1]), "v"(v[2]), "v"(v[3]), "v"(v[4]), "v"(v[5]), "v"(v[6]), "v"(v[7]), "s"(s)
               : "memory");
}
template <int CNT>
__device__ __forceinline__ void wait_unit(f32x4 (&b)[8]) {
  asm volatile("s_waitcnt vmcnt(%8)"
               : "+v"(b[0]), "+v"(b[1]), "+v"(b[2]), "+v"(b[3]), "+v"(b[4]), "+v"(b[5]), "+v"(b[6]), "+v"(b[7])
               : "i"(CNT) : "memory");
}
template <int CNT>
__device__ __forceinline__ void wait_vm() {
  asm volatile("s_waitcnt vmcnt(%0)" :: "i"(CNT) : "memory");
}
__device__ __forceinline__ void store_unit(const f32x4 (&b)[8], bool isw, unsigned v, char* d) {
  u32x4 o[4];
  if (isw) {
    for (int n = 0; n < 4; ++n)
      for (int e = 0; e < 4; ++e) o[n][e] = pk16(b[2 * e][n], b[2 * e + 1][n]);
  } else {
    for (int u = 0; u < 4; ++u) {
      o[u][0] = pk16(b[2 * u][0], b[2 * u][1]);
      o[u][1] = pk16(b[2 * u][2], b[2 * u][3]);
      o[u][2] = pk16(b[2 * u + 1][0], b[2 * u + 1][1]);
      o[u][3] = pk16(b[2 * u + 1][2], b[2 * u + 1][3]);
    }
  }
  asm volatile("global_store_dwordx4 %4, %0, %5 sc0 sc1\n\t"
               "global_store_dwordx4 %4, %1, %5 offset:1024 sc0 sc1\n\t"
               "global_store_dwordx4 %4, %2, %5 offset:2048 sc0 sc1\n\t"
               "global_store_dwordx4 %4, %3, %5 offset:3072 sc0 sc1\n\ts_nop 1"
               :: "v"(o[0]), "v"(o[1]), "v"(o[2]), "v"(o[3]), "v"(v), "s"(d) : "memory");
}
__device__ __forceinline__ void unit_offsets(unsigned (&v)[8], bool isw, int lane) {
  const int q = lane >> 4, c = lane & 15;
  for (int j = 0; j < 8; ++j)
    v[j] = isw ? (unsigned)((j >> 2) * 16 + 4 * q + (j & 3)) * (SHARD * 4) + c * 16
               : (unsigned)((j >> 2) * 16 + c) * (KDIM * 4) + (j & 3) * 64 + q * 16;
}
__device__ __forceinline__ unsigned w_unit(unsigned n) {
  return ((n >> 2) * 2 + (n & 1)) * (32 * SHARD * 4) + ((n >> 1) & 1) * 256;
}
__device__ __forceinline__ unsigned x_unit(unsigned n) {
  unsigned step, un;
  if (n < 512) { step = (n & 255) >> 2; un = (n >> 8) * 4 + (n & 3); }
  else if (n < 512 + 64 * U2) { step = (n - 512) / U2; un = 8 + (n - 512) % U2; }
  else { step = (n - 512 - 64 * U2) / UX; un = 8 + U2 + (n - 512 - 64 * U2) % UX; }
  return un * (32 * KDIM * 4) + step * 256;
}

template <bool CROSS>
__device__ __forceinline__ void push_stream(const char* wsrc, const char* xsrc, char* dstws, size_t dstoff,
                                            const unsigned* ctr, char* flag, int v, int lane,
                                            unsigned long long* dbg) {
  static_assert(SW >= 1 && SW <= 2, "window");
  constexpr int NIT = CROSS ? 4 + UX : 8 + U2;
  constexpr int STEADY = 8 * SW + 4 * (SW - 1);
  f32x4 buf[4][8];
  unsigned voff[8];
  unit_offsets(voff, CROSS, lane);
  static_for<4>([&](auto jc) {
    constexpr int j = decltype(jc)::value;
    const unsigned n = 2 * v + (j >> 1) * (2 * NWV) + (j & 1);
    load_unit(buf[j], voff, CROSS ? wsrc + w_unit(n) : xsrc + x_unit(n));
  });
  const unsigned call = *(const CONST_AS unsigned*)(uintptr_t)ctr + 1u;
  char* dst = dstws + OFF_DATA + (size_t)(call & 1) * PARITY_B + dstoff + (size_t)v * 8192;
  const unsigned vs = lane * 16;
#pragma unroll 1
  for (int it = 0; it < NIT; ++it) {
    const bool sw = CROSS && it < 4;
    const int nx = it + 1 < NIT ? it + 1 : it;
    const bool lw = CROSS && nx < 4;
    if (CROSS && it == 3) unit_offsets(voff, false, lane);
    static_for<4>([&](auto jc) {
      constexpr int j = decltype(jc)::value;
      if (it == 0) wait_vm<(3 - j) * 8 + j * 12>();
      wait_unit<STEADY>(buf[j]);
      if constexpr (((j - SW) & 1) != 0) {
        const int done = (4 * it + j - SW + 1) >> 1;
        if (done > 0 && lane == 0) flag_store(flag, call * 256u + (unsigned)done);
      }
      store_unit(buf[j], sw, vs, dst + (j >> 1) * (NWV * 8192) + (j & 1) * 4096);
      const unsigned n = nx * (4 * NWV) + 2 * v + (j >> 1) * (2 * NWV) + (j & 1);
      load_unit(buf[j], voff, lw ? wsrc + w_unit(n) : xsrc + x_unit(CROSS ? n + 256 + 64 * U2 : n));
    });
    dst += 2 * NWV * 8192;
  }
  BARRIER();
  wait_unit<0>(buf[0]);
  wait_unit<0>(buf[1]);
  wait_unit<0>(buf[2]);
  wait_unit<0>(buf[3]);
  if (lane == 0) flag_store(flag, call * 256u + (unsigned)(2 * NIT));
  STAMP(dbg, NIT);
}

__device__ __forceinline__ void convert_local(const char* src, bool isw, unsigned n0, char* dst, char* flag,
                                              unsigned call, int wave, int lane, unsigned long long* dbg) {
  if (wave < 8) {
    f32x4 buf[2][8];
    unsigned voff[8];
    unit_offsets(voff, isw, lane);
    const unsigned n = n0 + 2 * wave;
    if (CONV_DELAY > 0) __builtin_amdgcn_s_sleep(CONV_DELAY);
    load_unit(buf[0], voff, src + (isw ? w_unit(n & 255) : x_unit(n)));
    load_unit(buf[1], voff, src + (isw ? w_unit((n + 1) & 255) : x_unit(n + 1)));
    wait_unit<8>(buf[0]);
    store_unit(buf[0], isw, lane * 16, dst + (size_t)n * 4096);
    wait_unit<4>(buf[1]);
    store_unit(buf[1], isw, lane * 16, dst + (size_t)n * 4096 + 4096);
    wait_vm<0>();
  }
  BARRIER();
  if (wave == 9) {
    if (lane == 0) flag_store(flag, call * 256u + 1u);
    STAMP(dbg, 0);
  }
}

__device__ __forceinline__ int staged_row(int R, int c, int rank) {
  const int grp = rank >> 2, pos = rank & 3;
  if (R < 8 + U2) return rank * SHARD + R * 32;
  R -= 8 + U2;
  if (R < 24) {
    const int b = R >= 12 ? 1 : 0, m = R - 12 * b;
    return (grp * 4 + ((pos + 1 + (m >> 2)) & 3)) * SHARD + b * 128 + (m & 3) * 32;
  }
  R -= 24;
  const int k = R / UX;
  return ((grp ^ 1) * 4 + ((pos + c + 1 + k) & 3)) * SHARD + ROWX + (R - k * UX) * 32;
}

__device__ __forceinline__ void load_piece(u32x4 (&b)[8], unsigned v, const char* s) {
  asm volatile("global_load_dwordx4 %0, %8, %9 offset:-4096 sc0 sc1\n\t"
               "global_load_dwordx4 %1, %8, %9 offset:-3072 sc0 sc1\n\t"
               "global_load_dwordx4 %2, %8, %9 offset:-2048 sc0 sc1\n\t"
               "global_load_dwordx4 %3, %8, %9 offset:-1024 sc0 sc1\n\t"
               "global_load_dwordx4 %4, %8, %9 sc0 sc1\n\t"
               "global_load_dwordx4 %5, %8, %9 offset:1024 sc0 sc1\n\t"
               "global_load_dwordx4 %6, %8, %9 offset:2048 sc0 sc1\n\t"
               "global_load_dwordx4 %7, %8, %9 offset:3072 sc0 sc1"
               : "=&v"(b[0]), "=&v"(b[1]), "=&v"(b[2]), "=&v"(b[3]), "=&v"(b[4]), "=&v"(b[5]), "=&v"(b[6]), "=&v"(b[7])
               : "v"(v), "s"(s) : "memory");
}
__device__ __forceinline__ void store_rows(u32x4 a, u32x4 b, u32x4 c, u32x4 d, unsigned v, char* s) {
  asm volatile("global_store_dwordx4 %4, %0, %5 offset:-4096 sc0 sc1\n\t"
               "global_store_dwordx4 %4, %1, %5 offset:-2048 sc0 sc1\n\t"
               "global_store_dwordx4 %4, %2, %5 sc0 sc1\n\t"
               "global_store_dwordx4 %4, %3, %5 offset:2048 sc0 sc1\n\ts_nop 1"
               :: "v"(a), "v"(b), "v"(c), "v"(d), "v"(v), "s"(s) : "memory");
}

__device__ __forceinline__ void copy_stream(char* ws, char* out, char* oflag, int c, int rank, unsigned call,
                                            int v, int lane, unsigned long long* dbg) {
  const int cnt = (NSTG - v + NWV - 1) / NWV;
  const char* stg = ws + OFF_DATA + (size_t)(call & 1) * PARITY_B + D_STG + (size_t)c * MIB + v * 8192;
  const char* fp = ws + OFF_SFLAG + (size_t)(c * 128 + v + NWV * (lane < cnt ? lane : 0)) * 128;
  char* tmo = ws + OFF_TMO;
  const unsigned vl = 4096 + lane * 16;
  const unsigned vo = (unsigned)(4 * (lane >> 4) + 2) * (SHARD * 4) + (lane & 15) * 16;
  unsigned long long ready = 0;
  unsigned nanm = 0;
  auto wait_ready = [&](int i) {
    if ((ready >> i) & 1) return;
    const unsigned long long t0 = __builtin_amdgcn_s_memrealtime();
    for (;;) {
      unsigned f, tv;
      poll_once(f, tv, fp, tmo);
      ready = __ballot((int)(f - call) >= 0);
      if ((ready >> i) & 1) break;
      if (expired(tv, tmo, call, t0, lane)) { ready = ~0ull; nanm = 0x7fc00000u; break; }
      __builtin_amdgcn_s_sleep(8);
    }
  };
  u32x4 cur[8], nxt[8];
  wait_ready(0);
  load_piece(cur, vl, stg);
  asm volatile("s_waitcnt vmcnt(0)"
               : "+v"(cur[0]), "+v"(cur[1]), "+v"(cur[2]), "+v"(cur[3]), "+v"(cur[4]), "+v"(cur[5]), "+v"(cur[6]), "+v"(cur[7])
               :: "memory");
  wait_ready(1);
  BARRIER();
  STAMP(dbg, 0);
#pragma unroll 1
  for (int i = 0; i < cnt; ++i) {
    const int nx = i + 1 < cnt ? i + 1 : i;
    wait_ready(nx);
    const int e = v + NWV * i;
    char* d = out + (size_t)staged_row(e >> 1, c, rank) * (SHARD * 4) + (e & 1) * 256;
    for (int k = 0; k < 8; ++k) cur[k] |= nanm;
    store_rows(cur[0], cur[1], cur[2], cur[3], vo, d);
    load_piece(nxt, vl, stg + nx * (NWV * 8192));
    unsigned f, tv;
    asm volatile("global_load_dword %0, %2, off sc0 sc1\n\t"
                 "global_load_dword %1, %3, %4 sc1\n\ts_waitcnt vmcnt(%5)"
                 : "=&v"(f), "=&v"(tv) : "v"(fp), "v"(0u), "s"(tmo), "i"(SW == 1 ? 10 : 14) : "memory");
    store_rows(cur[4], cur[5], cur[6], cur[7], vo + 16 * (SHARD * 4), d);
    asm volatile("s_waitcnt vmcnt(%10)"
                 : "+v"(nxt[0]), "+v"(nxt[1]), "+v"(nxt[2]), "+v"(nxt[3]), "+v"(nxt[4]), "+v"(nxt[5]), "+v"(nxt[6]),
                   "+v"(nxt[7]), "+v"(f), "+v"(tv) : "i"(SW == 1 ? 0 : 4) : "memory");
    ready |= __ballot((int)(f - call) >= 0);
    if (__builtin_amdgcn_readfirstlane(tv) == call) nanm = 0x7fc00000u;
    for (int k = 0; k < 8; ++k) cur[k] = nxt[k];
    STAMP(dbg, 1 + i);
  }
  wait_vm<0>();
  if (lane == 0) flag_store(oflag, call);
}

__device__ __forceinline__ float gelu(float y) {
  const float k1 = -2.0f * 0.7978845608028654f * 1.4426950408889634f;
  const float k3 = k1 * 0.044715f;
  float e = __builtin_amdgcn_exp2f(y * (k1 + k3 * y * y));
  return y * __builtin_amdgcn_rcpf(1.0f + e);
}

__host__ __device__ constexpr int ring_slots(int nu) { return nu == 5 ? 5 : nu == 4 ? 6 : nu == 3 ? 7 : 8; }

struct Tile {
  unsigned a, astep, b;
  unsigned aflag, bflag;
  int acount, abase, au;
  bool alocal, blocal;
  int nu;
  char* out;
  char* oflag;
  bool staged;
};

template <int NU>
__device__ __forceinline__ void tile_loader(unsigned lbase, const char* a, unsigned astep, const char* b, int j,
                                            int lane) {
  constexpr int SLOT = NU * 4096 + 8192;
  constexpr int NS = ring_slots(NU);
  constexpr int D = NS - 1;
  constexpr int NLD = NU + 2;
  static_assert(NS * SLOT <= LDS_RING && (D - 1) * NLD <= 63, "ring");
  const unsigned voff = lane * 16;
  a += j * (NU * 1024);
  b += j * 2048;
  unsigned slot = 0;
#pragma unroll 1
  for (int t = 0; t < NSTEP; ++t) {
    BARRIER();
    const unsigned la = lbase + slot + j * (NU * 1024);
    const unsigned lb = lbase + slot + NU * 4096 + j * 2048;
    unsigned keep;
#define AG_A(off) "global_load_lds_dwordx4 %1, %2 offset:" #off " sc0 sc1\n\t"
#define AG_HEAD "s_mov_b32 %0, m0\n\ts_mov_b32 m0, %4\n\ts_nop 0\n\t"
#define AG_TAIL "s_mov_b32 m0, %5\n\ts_nop 0\n\t" \
                "global_load_lds_dwordx4 %1, %3 sc0 sc1\n\t" \
                "global_load_lds_dwordx4 %1, %3 offset:1024 sc0 sc1\n\t" \
                "s_mov_b32 m0, %0\n\ts_waitcnt vmcnt(%6)"
#define AG_OPS : "=&s"(keep) : "v"(voff), "s"(a), "s"(b), "s"(la), "s"(lb), "i"((D - 1) * NLD), \
                 "v"(voff + 4096), "s"(la + 4096) : "memory"
    if constexpr (NU == 1) asm volatile(AG_HEAD AG_A(0) AG_TAIL AG_OPS);
    if constexpr (NU == 2) asm volatile(AG_HEAD AG_A(0) AG_A(1024) AG_TAIL AG_OPS);
    if constexpr (NU == 3) asm volatile(AG_HEAD AG_A(0) AG_A(1024) AG_A(2048) AG_TAIL AG_OPS);
    if constexpr (NU == 4) asm volatile(AG_HEAD AG_A(0) AG_A(1024) AG_A(2048) AG_A(3072) AG_TAIL AG_OPS);
    if constexpr (NU == 5)
      asm volatile(AG_HEAD AG_A(0) AG_A(1024) AG_A(2048) AG_A(3072)
                   "s_mov_b32 m0, %8\n\ts_nop 0\n\tglobal_load_lds_dwordx4 %7, %2 sc0 sc1\n\t" AG_TAIL AG_OPS);
    a += astep;
    b += 16384;
    slot += SLOT;
    if (slot == NS * SLOT) slot = 0;
  }
  static_for<D>([&](auto tc) {
    constexpr int t = decltype(tc)::value;
    BARRIER();
    wait_vm<(D - 2 - t > 0 ? D - 2 - t : 0) * NLD>();
  });
  BARRIER();
}

__device__ __forceinline__ void gemm_tile(LDS_AS char* smem, char* ws, const Tile& T, unsigned call,
                                          int wave, int lane, unsigned long long* dbg) {
  const int nu = T.nu;
  const int D = ring_slots(nu) - 1;
  LDS_AS unsigned* bad = (LDS_AS unsigned*)(smem + LDS_RING);

  if (wave == 9) {
    const bool lo = lane < 32;
    const int cnt = lo ? T.acount : 16;
    const int g = (lane & 31) < cnt ? (lane & 31) : cnt - 1;
    const bool loc = lo ? T.alocal : T.blocal;
    const char* fp = ws + (lo ? T.aflag : T.bflag) + g * FLAG_STRIDE;
    const int mul = loc ? 4096 : 2 * NWV;
    const int add = g * (loc ? 16 : 2) - (lo ? T.abase : 0);
    const int per = lo ? T.au : 4;
    char* tmo = ws + OFF_TMO;
    int first = -1;
    int need = per;
    bool dead = false;
    STAMP(dbg, 0);
    for (int t = 0; t < NSTEP; ++t) {
      if (!dead && !__all(first >= need)) {
        const unsigned long long t0 = __builtin_amdgcn_s_memrealtime();
        for (;;) {
          unsigned f, tv;
          poll_once(f, tv, fp, tmo);
          const int d = (int)(f - call * 256u);
          first = (d > 0 ? d : 0) * mul + add;
          if (__all(first >= need)) break;
          if (expired(tv, tmo, call, t0, lane)) { dead = true; break; }
          __builtin_amdgcn_s_sleep(POLL_SLEEP);
        }
      }
      need += per;
      BARRIER();
    }
    {
      const unsigned tv = __hip_atomic_load((GLOBAL_AS unsigned*)tmo, __ATOMIC_RELAXED, __HIP_MEMORY_SCOPE_AGENT);
      if (lane == 0) *bad = (dead || tv == call) ? 0x7fc00000u : 0u;
    }
    for (int t = 0; t <= D; ++t) BARRIER();
    STAMP(dbg, 4);
  } else if (wave >= 5) {
    const char* data = ws + OFF_DATA + (size_t)(call & 1) * PARITY_B;
    const unsigned lbase = (unsigned)(uintptr_t)smem;
    const char* a = data + T.a;
    const char* b = data + T.b;
    const int j = wave - 5;
    if (nu == 4) tile_loader<4>(lbase, a, T.astep, b, j, lane);
    else if (nu == (U2 == 4 ? 2 : 1)) tile_loader<U2 == 4 ? 2 : 1>(lbase, a, T.astep, b, j, lane);
    else if (nu == 3) tile_loader<3>(lbase, a, T.astep, b, j, lane);
    else tile_loader<5>(lbase, a, T.astep, b, j, lane);
  } else if (wave >= nu) {
    for (int t = 0; t < NSTEP + D + 1; ++t) BARRIER();
  } else {
    f32x4 acc[2][4];
    for (int m = 0; m < 2; ++m)
      for (int n = 0; n < 4; ++n) acc[m][n] = f32x4{0.f, 0.f, 0.f, 0.f};
    f16x8 fa[2][2][2], fb[2][4][2];
    LDS_AS char* pa = smem + wave * 4096 + lane * 16;
    LDS_AS char* pb = smem + nu * 4096 + lane * 16;
    const unsigned sstep = nu * 4096 + 8192, send = sstep * (D + 1);
    unsigned slot = 0;
    auto rd = [&](auto sc) {
      constexpr int s = decltype(sc)::value;
      for (int u = 0; u < 2; ++u) {
        for (int m = 0; m < 2; ++m) fa[s][m][u] = *(LDS_AS f16x8*)(pa + slot + m * 2048 + u * 1024);
        for (int n = 0; n < 4; ++n) fb[s][n][u] = *(LDS_AS f16x8*)(pb + slot + u * 4096 + n * 1024);
      }
      slot += sstep;
      if (slot == send) slot = 0;
    };
    auto mm = [&](auto sc) {
      constexpr int s = decltype(sc)::value;
      for (int u = 0; u < 2; ++u)
        for (int m = 0; m < 2; ++m)
          for (int n = 0; n < 4; ++n)
            acc[m][n] = __builtin_amdgcn_mfma_f32_16x16x32_f16(fa[s][m][u], fb[s][n][u], acc[m][n], 0, 0, 0);
    };
    using I0 = std::integral_constant<int, 0>;
    using I1 = std::integral_constant<int, 1>;
    for (int t = 0; t <= D; ++t) BARRIER();
    rd(I0{});
    for (int s = 1; s < NSTEP - 1; s += 2) {
      BARRIER();
      rd(I1{});
      __builtin_amdgcn_sched_barrier(0);
      mm(I0{});
      __builtin_amdgcn_sched_barrier(0);
      BARRIER();
      rd(I0{});
      __builtin_amdgcn_sched_barrier(0);
      mm(I1{});
      __builtin_amdgcn_sched_barrier(0);
    }
    BARRIER();
    rd(I1{});
    const unsigned isbad = *bad;
    __builtin_amdgcn_sched_barrier(0);
    mm(I0{});
    __builtin_amdgcn_sched_barrier(0);
    BARRIER();
    mm(I1{});
    char* out = T.out + (size_t)wave * (T.staged ? 2 * 8192 : 32 * SHARD * 4);
    const unsigned vo = T.staged ? lane * 16 : (unsigned)(4 * (lane >> 4)) * (SHARD * 4) + (lane & 15) * 16;
    const unsigned sj = T.staged ? 1024 : SHARD * 4, sm = T.staged ? 4096 : 16 * SHARD * 4;
    for (int m = 0; m < 2; ++m) {
      for (int jj = 0; jj < 4; ++jj) {
        u32x4 o;
        for (int n = 0; n < 4; ++n)
          o[n] = isbad | __builtin_bit_cast(unsigned, gelu(acc[m][n][jj]));
        asm volatile("global_store_dwordx4 %1, %0, %2 sc0 sc1\n\ts_nop 1"
                     :: "v"(o), "v"(vo + m * sm + jj * sj), "s"(out) : "memory");
      }
    }
    if (T.oflag != nullptr) {
      wait_vm<0>();
      if (lane == 0) flag_store(T.oflag + wave * 256, call);
    }
    if (wave == 0) STAMP(dbg, 5);
  }
}

__device__ __forceinline__ Tile make_tile(const Params& P, int src, int u0, int nu, bool foreign, int xcd,
                                          unsigned call) {
  const int grp = P.rank >> 2, pos = P.rank & 3;
  const int c = xcd >> 1, ch = xcd & 1, cr = (4 - c) & 3;
  Tile T;
  int row0, R;
  bool direct = false;
  if (src < 4) {
    const int ub = src < 3 ? 256 * src : 512 + 64 * U2;
    T.au = src < 2 ? 4 : src == 2 ? U2 : UX;
    T.a = D_OWNX + ub * 4096;
    T.aflag = OFF_FLAG + (ub >> 4) * FLAG_STRIDE; T.acount = 4 * T.au; T.abase = 0; T.alocal = true;
    row0 = P.rank * SHARD + (ub >> 1);
    R = ub >> 6;
  } else if (src < 13) {
    const int i = (src - 4) / 3, b = src - 4 - 3 * i;
    T.au = b < 2 ? 4 : U2;
    T.a = D_INTRA + (3 * i + b) * MIB;
    T.aflag = OFF_FLAG + (8 + i) * 2048; T.acount = NWV; T.abase = 256 * b; T.alocal = false;
    row0 = (grp * 4 + ((pos + i + 1) & 3)) * SHARD + b * 128;
    R = b < 2 ? 8 + U2 + 12 * b + 4 * i : U2 * i;
    direct = b == 2;
  } else {
    const int k = src - 13;
    T.au = UX;
    T.a = D_CROSS + (3 * k + 1) * MIB;
    T.aflag = OFF_FLAG + (12 + k) * 2048; T.acount = NWV; T.abase = 256; T.alocal = false;
    row0 = ((grp ^ 1) * 4 + ((pos + k) & 3)) * SHARD + ROWX;
    R = 32 + U2 + ((k - c - 1) & 3) * UX;
  }
  T.nu = nu == 0 ? T.au : nu;
  T.astep = T.au * 4096;
  T.a += u0 * 4096;
  row0 += u0 * 32;
  R += u0;
  T.staged = foreign && !direct;
  T.blocal = !foreign;
  if (!foreign) {
    T.b = D_OWNW + c * MIB + ch * 8192;
    T.bflag = OFF_FLAG + 4 * 2048 + c * 16 * FLAG_STRIDE;
    T.out = (char*)P.out[P.rank & 7] + (size_t)row0 * (SHARD * 4) + xcd * 256;
    T.oflag = nullptr;
  } else {
    T.b = D_CROSS + 3 * c * MIB + ch * 8192;
    T.bflag = OFF_FLAG + (12 + c) * 2048;
    if (direct) {
      const int drank = ((grp ^ 1) * 4 + ((pos + c) & 3)) & 7;
      T.out = (char*)P.out[drank] + (size_t)row0 * (SHARD * 4) + cr * 512 + ch * 256;
      T.oflag = P.ws[drank] + OFF_OFLAG + (size_t)(cr * 64 + 16 + 2 * R + ch) * 128;
    } else {
      T.out = P.self + OFF_DATA + (size_t)(call & 1) * PARITY_B + D_STG + c * MIB + (2 * R + ch) * 8192;
      T.oflag = P.self + OFF_SFLAG + (size_t)(c * 128 + 2 * R + ch) * 128;
    }
  }
  return T;
}

__global__ void __launch_bounds__(NTHREADS, 1) ag_gemm_kernel(Params P) {
  __shared__ __attribute__((aligned(16))) char smem_raw[LDS_BYTES];
  LDS_AS char* smem = (LDS_AS char*)smem_raw;
  const int lane = threadIdx.x & 63;
  const int wave = __builtin_amdgcn_readfirstlane(threadIdx.x >> 6);
  const int bx = blockIdx.x, xcd = bx & 7, j = bx >> 3;
  const int grp = P.rank >> 2, pos = P.rank & 3;
  char* ws = P.self;
  unsigned* ctr = (unsigned*)(ws + OFF_CTR + (size_t)bx * 128);
  unsigned long long* dbg = (unsigned long long*)(ws + OFF_DBG + (size_t)bx * 512);
  if (wave == 9) STAMP(dbg, 0);

  const int link = PMAP == 0 ? j : (j < 2 ? 3 + 2 * j + (xcd >> 2) : 2 * (j - 2) + (xcd >> 2) + (j == 3 ? 4 * (xcd >> 2) : 0));
  const int sub = PMAP == 0 ? ((xcd + j) & 7) : (xcd & 3);
  if (j < JP && link >= 7) return;
  if (j < JP && wave < PW) {
    const int v = sub * PW + wave;
    if (link < 3) {
      const int i = link + 1, ir = 4 - i;
      char* pw = P.ws[(grp * 4 + ((pos + i) & 3)) & 7];
      push_stream<false>(nullptr, (const char*)P.x, pw, D_INTRA + (size_t)(ir - 1) * 3 * MIB, ctr,
                         pw + OFF_FLAG + (7 + ir) * 2048 + v * FLAG_STRIDE, v, lane, dbg + (wave & 1) * 16);
    } else {
      const int c = link - 3, cr = (4 - c) & 3;
      char* pw = P.ws[((grp ^ 1) * 4 + ((pos + c) & 3)) & 7];
      push_stream<true>((const char*)P.w + c * 512, (const char*)P.x, pw, D_CROSS + (size_t)cr * 3 * MIB, ctr,
                        pw + OFF_FLAG + (12 + cr) * 2048 + v * FLAG_STRIDE, v, lane, dbg + (wave & 1) * 16);
    }
    return;
  }
  if (j < JP && wave < 9 && (link < 3 || wave >= 2 * PW)) return;
  const unsigned call = *(const CONST_AS unsigned*)(uintptr_t)ctr + 1u;
  if (j < JP) {
    const int c = link - 3, cr = (4 - c) & 3;
    if (wave < 9) {
      const int v = sub * PW + wave - PW;
      const int drank = ((grp ^ 1) * 4 + ((pos + c) & 3)) & 7;
      copy_stream(ws, (char*)P.out[drank] + cr * 512, P.ws[drank] + OFF_OFLAG + (size_t)(cr * 64 + v) * 128, c, P.rank,
                  call, v, lane, dbg + 32 + (wave & 1) * 12);
      return;
    }
    BARRIER();
    if (link >= 3 && sub == 0) {
      const char* fp = ws + OFF_OFLAG + (size_t)(c * 64 + (lane < 16 + NDIR ? lane : 0)) * 128;
      char* tmo = ws + OFF_TMO;
      const unsigned long long t0 = __builtin_amdgcn_s_memrealtime();
      for (;;) {
        unsigned f, tv;
        poll_once(f, tv, fp, tmo);
        if (__all((int)(f - call) >= 0)) break;
        if (expired(tv, tmo, call, t0, lane)) {
          if (lane == 0) flag_store(P.out[P.rank & 7] + (size_t)P.rank * SHARD * SHARD, 0x7fc00000u);
          break;
        }
        __builtin_amdgcn_s_sleep(2);
      }
      STAMP(dbg, 62);
    }
  } else {
    const int r = j - JP;
    constexpr int NT2 = U2 == 4 ? 12 : 6 * U2;
    int s0 = -1, s1 = -1, t2 = -1;
    bool f0 = false;
    if (r < 3) { s0 = r; f0 = true; }
    else if (r < 9) { s0 = 4 + 3 * ((r - 3) >> 1); f0 = r & 1; }
    else if (r < 15) { s0 = 5 + 3 * ((r - 9) >> 1); f0 = r & 1; }
    else if (r < 19) { s0 = 13 + r - 15; }
    else if (r < 22) { s0 = 13 + (((xcd >> 1) + 1 + r - 19) & 3); f0 = true; }
    else if (r < 26) { s0 = r - 22; if (JP == 7 && r == 24) s1 = 3; }
    if (r < 9) t2 = r;
    else if (r >= 22) t2 = r - 13;
    else if (r < 15) t2 = r + (32 - JP) - 22;
    if (r >= 9 && r < 25) {
      const int q = (r - 9) * 8 + xcd;
      const bool isw = q >= 64;
      convert_local(isw ? (const char*)P.w + ((q - 64) >> 4) * 512 : (const char*)P.x, isw, q * 16,
                    ws + OFF_DATA + (size_t)(call & 1) * PARITY_B,
                    ws + OFF_FLAG + (isw ? 4 * 2048 + (q - 64) * FLAG_STRIDE : q * FLAG_STRIDE), call, wave, lane, dbg + 40);
    }
#pragma unroll 1
    for (int n = 0; n < 3; ++n) {
      int src = n == 0 ? s0 : s1, u0 = 0, nu = 0;
      bool foreign = n == 0 && f0;
      if (n == 2) {
        constexpr int PER = NT2 / 3;
        const int i = t2 / PER, m = t2 - i * PER;
        src = t2 >= 0 && t2 < NT2 ? 6 + 3 * i : -1;
        nu = U2 == 4 ? 2 : 1;
        u0 = (m >> 1) * nu;
        foreign = m & 1;
      }
      if (src < 0) continue;
      const Tile T = make_tile(P, src, u0, nu, foreign, xcd, call);
      gemm_tile(smem, ws, T, call, wave, lane, dbg + 8 + 8 * n);
    }
  }
  if (wave == 9) {
    if (lane == 0) __hip_atomic_store((GLOBAL_AS unsigned*)ctr, call, __ATOMIC_RELAXED, __HIP_MEMORY_SCOPE_AGENT);
    STAMP(dbg, 63);
  }
}

extern "C" void kernel_launch(int rank, int nranks, ncclComm_t comm,
                              hipStream_t stream,
                              void* const* d_in, const int* in_sizes, int n_in,
                              void* d_out, int out_size,
                              const int* in_dtypes, int out_dtype,
                              void* const* d_out_all,
                              void* const* d_ws_all, size_t ws_bytes) {
  (void)comm; (void)in_dtypes; (void)out_dtype; (void)d_out;
  if (nranks != NRANK || (unsigned)rank >= (unsigned)NRANK || n_in < 2 || ws_bytes < WS_NEED ||
      in_sizes[0] != SHARD * KDIM || in_sizes[1] != KDIM * SHARD || out_size != NRANK * SHARD * SHARD) {
    fprintf(stderr, "unexpected problem shape\n");
    return;
  }
  Params P;
  P.x = (const float*)d_in[0];
  P.w = (const float*)d_in[1];
  for (int p = 0; p < NRANK; ++p) {
    P.ws[p] = (char*)d_ws_all[p];
    P.out[p] = (float*)d_out_all[p];
  }
  P.self = (char*)d_ws_all[rank];
  P.rank = rank;
  P.pad = 0;
  ag_gemm_kernel<<<NWG, NTHREADS, 0, stream>>>(P);
  HIP_CHECK(hipGetLastError());
}
```

```cpp
#include <hip/hip_runtime.h>
#include <rccl/rccl.h>
#include <cstdio>
#include <cstdint>
#include <utility>

#define HIP_CHECK(expr) do { hipError_t _e = (expr); if (_e != hipSuccess) { \
    fprintf(stderr, "HIP error %s at %s:%d\n", hipGetErrorString(_e), __FILE__, __LINE__); \
    return; } } while (0)

typedef unsigned u32x4 __attribute__((ext_vector_type(4)));
typedef float f32x4 __attribute__((ext_vector_type(4)));
typedef _Float16 f16x8 __attribute__((ext_vector_type(8)));
#define LDS_AS __attribute__((address_space(3)))
#define CONST_AS __attribute__((address_space(4)))
#define GLOBAL_AS __attribute__((address_space(1)))

#define U2 3
#define PW 2
#define BLAST_C 1
#define HO_EARLY 0
#define FLAG_STRIDE 128
#define POLL_SLEEP 4
#define CONV_DELAY 0

constexpr int NRANK = 8;
constexpr int KDIM = 4096;
constexpr int SHARD = 512;
constexpr int UX = 8 - U2;
constexpr int ROWX = 256 + 32 * U2;
constexpr int NWV = 8 * PW;
constexpr int JP = 7;
constexpr int NWG = 256;
constexpr int NTHREADS = 640;
constexpr unsigned MIB = 1u << 20;
constexpr int NSTEP = 64;
constexpr int NSTG = 2 * (32 + U2 + 3 * UX);
constexpr int NDIR = 2 * 3 * U2;

constexpr size_t OFF_CTR = 0;
constexpr size_t OFF_TMO = 32768;
constexpr size_t OFF_TICK = 36864;
constexpr size_t OFF_SFLAG = 49152;
constexpr size_t OFF_FLAG = 65536;
constexpr size_t OFF_OFLAG = 131072;
constexpr size_t OFF_DBG = 262144;
constexpr size_t OFF_DATA = MIB;
constexpr size_t PARITY_B = 40 * MIB;
constexpr size_t D_OWNX = 0, D_OWNW = 4 * MIB, D_INTRA = 8 * MIB, D_CROSS = 17 * MIB, D_STG = 29 * MIB;
constexpr size_t WS_NEED = OFF_DATA + 2 * PARITY_B;
static_assert(NWV * FLAG_STRIDE <= 4096 && NWV <= 32 && NSTG <= 128 && NWV + NDIR <= 64, "flag blocks");
constexpr unsigned FSEQ = 1024;

constexpr unsigned long long DEADLINE_TICKS = 40000000ull;

constexpr int LDS_RING = 147456;
constexpr int LDS_BYTES = LDS_RING + 64;

struct Params {
  const float* x;
  const float* w;
  char* self;
  char* ws[NRANK];
  float* out[NRANK];
  int rank;
  int pad;
};

template <int... I, class F>
__device__ __forceinline__ void static_for_impl(std::integer_sequence<int, I...>, F&& f) {
  (f(std::integral_constant<int, I>{}), ...);
}
template <int N, class F>
__device__ __forceinline__ void static_for(F&& f) {
  static_for_impl(std::make_integer_sequence<int, N>{}, f);
}

#define BARRIER() do { asm volatile("" ::: "memory"); __builtin_amdgcn_s_waitcnt(0xC07F); \
    asm volatile("s_barrier" ::: "memory"); } while (0)

#define STAMP(dbg, id) do { } while (0)

__device__ __forceinline__ unsigned pk16(float a, float b) {
  typedef _Float16 f16x2 __attribute__((ext_vector_type(2)));
  f16x2 h;
  h[0] = (_Float16)a;
  h[1] = (_Float16)b;
  return __builtin_bit_cast(unsigned, h);
}

__device__ __forceinline__ void flag_store(void* p, unsigned v) {
  __hip_atomic_store((GLOBAL_AS unsigned*)p, v, __ATOMIC_RELAXED, __HIP_MEMORY_SCOPE_SYSTEM);
}
__device__ __forceinline__ void poll_once(unsigned& f, unsigned& tv, const char* fp, const char* tmo) {
  asm volatile("global_load_dword %0, %2, off sc0 sc1\n\t"
               "global_load_dword %1, %3, %4 sc1\n\ts_waitcnt vmcnt(0)"
               : "=&v"(f), "=&v"(tv) : "v"(fp), "v"(0u), "s"(tmo) : "memory");
}
__device__ __forceinline__ bool expired(unsigned tv, char* tmo, unsigned call, unsigned long long t0, int lane) {
  if (__builtin_amdgcn_readfirstlane(tv) == call || __builtin_amdgcn_s_memrealtime() - t0 > DEADLINE_TICKS) {
    if (lane == 0) __hip_atomic_store((GLOBAL_AS unsigned*)tmo, call, __ATOMIC_RELAXED, __HIP_MEMORY_SCOPE_AGENT);
    return true;
  }
  return false;
}

__device__ __forceinline__ void load_unit(f32x4 (&b)[8], const unsigned (&v)[8], const char* s) {
  asm volatile("global_load_dwordx4 %0, %8, %16\n\t"
               "global_load_dwordx4 %1, %9, %16\n\t"
               "global_load_dwordx4 %2, %10, %16\n\t"
               "global_load_dwordx4 %3, %11, %16\n\t"
               "global_load_dwordx4 %4, %12, %16\n\t"
               "global_load_dwordx4 %5, %13, %16\n\t"
               "global_load_dwordx4 %6, %14, %16\n\t"
               "global_load_dwordx4 %7, %15, %16"
               : "=&v"(b[0]), "=&v"(b[1]), "=&v"(b[2]), "=&v"(b[3]), "=&v"(b[4]), "=&v"(b[5]), "=&v"(b[6]), "=&v"(b[7])
               : "v"(v[0]), "v"(v[1]), "v"(v[2]), "v"(v[3]), "v"(v[4]), "v"(v[5]), "v"(v[6]), "v"(v[7]), "s"(s)
               : "memory");
}
template <int CNT>
__device__ __forceinline__ void wait_unit(f32x4 (&b)[8]) {
  asm volatile("s_waitcnt vmcnt(%8)"
               : "+v"(b[0]), "+v"(b[1]), "+v"(b[2]), "+v"(b[3]), "+v"(b[4]), "+v"(b[5]), "+v"(b[6]), "+v"(b[7])
               : "i"(CNT) : "memory");
}
template <int CNT>
__device__ __forceinline__ void wait_vm() {
  asm volatile("s_waitcnt vmcnt(%0)" :: "i"(CNT) : "memory");
}
__device__ __forceinline__ void store_unit(const f32x4 (&b)[8], bool isw, unsigned v, char* d) {
  u32x4 o[4];
  if (isw) {
    for (int n = 0; n < 4; ++n)
      for (int e = 0; e < 4; ++e) o[n][e] = pk16(b[2 * e][n], b[2 * e + 1][n]);
  } else {
    for (int u = 0; u < 4; ++u) {
      o[u][0] = pk16(b[2 * u][0], b[2 * u][1]);
      o[u][1] = pk16(b[2 * u][2], b[2 * u][3]);
      o[u][2] = pk16(b[2 * u + 1][0], b[2 * u + 1][1]);
      o[u][3] = pk16(b[2 * u + 1][2], b[2 * u + 1][3]);
    }
  }
  asm volatile("global_store_dwordx4 %4, %0, %5 sc0 sc1\n\t"
               "global_store_dwordx4 %4, %1, %5 offset:1024 sc0 sc1\n\t"
               "global_store_dwordx4 %4, %2, %5 offset:2048 sc0 sc1\n\t"
               "global_store_dwordx4 %4, %3, %5 offset:3072 sc0 sc1\n\ts_nop 1"
               :: "v"(o[0]), "v"(o[1]), "v"(o[2]), "v"(o[3]), "v"(v), "s"(d) : "memory");
}
__device__ __forceinline__ void unit_offsets(unsigned (&v)[8], bool isw, int lane) {
  const int q = lane >> 4, c = lane & 15;
  for (int j = 0; j < 8; ++j)
    v[j] = isw ? (unsigned)((j >> 2) * 16 + 4 * q + (j & 3)) * (SHARD * 4) + c * 16
               : (unsigned)((j >> 2) * 16 + c) * (KDIM * 4) + (j & 3) * 64 + q * 16;
}
__device__ __forceinline__ unsigned w_unit(unsigned n) {
  return ((n >> 2) * 2 + (n & 1)) * (32 * SHARD * 4) + ((n >> 1) & 1) * 256;
}
__device__ __forceinline__ unsigned x_unit(unsigned n) {
  unsigned step, un;
  if (n < 512) { step = (n & 255) >> 2; un = (n >> 8) * 4 + (n & 3); }
  else if (n < 512 + 64 * U2) { step = (n - 512) / U2; un = 8 + (n - 512) % U2; }
  else { step = (n - 512 - 64 * U2) / UX; un = 8 + U2 + (n - 512 - 64 * U2) % UX; }
  return un * (32 * KDIM * 4) + step * 256;
}

__device__ __forceinline__ void draw(unsigned& tk, const char* tick, unsigned inc) {
  asm volatile("s_mov_b64 exec, 1\n\t"
               "global_atomic_add %0, %1, %2, %3 sc0\n\t"
               "s_mov_b64 exec, -1"
               : "=&v"(tk) : "v"(0u), "v"(inc), "s"(tick) : "memory");
}

template <bool CROSS>
__device__ __forceinline__ void push_stream(const char* wsrc, const char* xsrc, char* dstws, size_t dstoff,
                                            const unsigned* ctr, const char* tick, char* flag, LDS_AS unsigned* ho,
                                            int v, int lane, unsigned long long* dbg) {
  constexpr unsigned NCH = 32 * (CROSS ? 4 + UX : 8 + U2);
  f32x4 b0[8], b1[8];
  unsigned voff[8];
  unit_offsets(voff, CROSS, lane);
  auto src = [&](unsigned n) {
    return CROSS ? (n < 256 ? wsrc + w_unit(n) : xsrc + x_unit(n + 256 + 64 * U2)) : xsrc + x_unit(n);
  };
  __builtin_amdgcn_s_waitcnt(0x0F70);
  load_unit(b0, voff, src(2 * v));
  load_unit(b1, voff, src(2 * v + 1));
  const unsigned call = *(const CONST_AS unsigned*)(uintptr_t)ctr + 1u;
  asm volatile("" :: "s"(call));
  BARRIER();
  char* dst = dstws + OFF_DATA + (size_t)(call & 1) * PARITY_B + dstoff;
  const unsigned vs = lane * 16;
  const unsigned base = (call - 1) * (NCH - NWV) - 2 * NWV;
  unsigned cur = v, nxt = NWV + v;
  bool vw = CROSS;
  wait_unit<8>(b0);
  STAMP(dbg, 15);
  bool last;
#pragma unroll 1
  do {
    last = nxt >= NCH;
    if (last) nxt = NCH - 1;
    store_unit(b0, CROSS && cur < 128, vs, dst + cur * 8192);
    if (CROSS && vw && nxt >= 128) { unit_offsets(voff, false, lane); vw = false; }
    unsigned tk;
    draw(tk, tick, last ? 0u : 1u);
    load_unit(b0, voff, src(2 * nxt));
    wait_unit<9>(b1);
    store_unit(b1, CROSS && cur < 128, vs, dst + cur * 8192 + 4096);
    if (CROSS && HO_EARLY && last) *(volatile LDS_AS unsigned*)ho = 1;
    load_unit(b1, voff, src(2 * nxt + 1));
    wait_unit<8>(b0);
    asm volatile("" : "+v"(tk));
    STAMP(dbg, 1 + (cur >> 5));
    cur = nxt;
    nxt = __builtin_amdgcn_readfirstlane(tk) - base;
    if (lane == 0) flag_store(flag, call * FSEQ + (last ? FSEQ - 1 : cur));
  } while (!last);
  if (CROSS && !HO_EARLY) *(volatile LDS_AS unsigned*)ho = 1;
  wait_unit<0>(b1);
}

__device__ __forceinline__ void convert_local(const char* src, bool isw, unsigned n0, char* dst, char* flag,
                                              unsigned call, int wave, int lane, unsigned long long* dbg) {
  if (wave < 8) {
    f32x4 buf[2][8];
    unsigned voff[8];
    unit_offsets(voff, isw, lane);
    const unsigned n = n0 + 2 * wave;
    if (CONV_DELAY > 0) __builtin_amdgcn_s_sleep(CONV_DELAY);
    load_unit(buf[0], voff, src + (isw ? w_unit(n & 255) : x_unit(n)));
    load_unit(buf[1], voff, src + (isw ? w_unit((n + 1) & 255) : x_unit(n + 1)));
    wait_unit<8>(buf[0]);
    store_unit(buf[0], isw, lane * 16, dst + (size_t)n * 4096);
    wait_unit<4>(buf[1]);
    store_unit(buf[1], isw, lane * 16, dst + (size_t)n * 4096 + 4096);
    wait_vm<0>();
  }
  BARRIER();
  if (wave == 9) {
    if (lane == 0) flag_store(flag, call * FSEQ + 1u);
    STAMP(dbg, 0);
  }
}

__device__ __forceinline__ int staged_row(int R, int c, int rank) {
  const int grp = rank >> 2, pos = rank & 3;
  if (R < 8 + U2) return rank * SHARD + R * 32;
  R -= 8 + U2;
  if (R < 24) {
    const int b = R >= 12 ? 1 : 0, m = R - 12 * b;
    return (grp * 4 + ((pos + 1 + (m >> 2)) & 3)) * SHARD + b * 128 + (m & 3) * 32;
  }
  R -= 24;
  const int k = R / UX;
  return ((grp ^ 1) * 4 + ((pos + c + 1 + k) & 3)) * SHARD + ROWX + (R - k * UX) * 32;
}

__device__ __forceinline__ void load_piece(u32x4 (&b)[8], unsigned v, const char* s) {
  asm volatile("global_load_dwordx4 %0, %8, %9 offset:-4096 sc0 sc1\n\t"
               "global_load_dwordx4 %1, %8, %9 offset:-3072 sc0 sc1\n\t"
               "global_load_dwordx4 %2, %8, %9 offset:-2048 sc0 sc1\n\t"
               "global_load_dwordx4 %3, %8, %9 offset:-1024 sc0 sc1\n\t"
               "global_load_dwordx4 %4, %8, %9 sc0 sc1\n\t"
               "global_load_dwordx4 %5, %8, %9 offset:1024 sc0 sc1\n\t"
               "global_load_dwordx4 %6, %8, %9 offset:2048 sc0 sc1\n\t"
               "global_load_dwordx4 %7, %8, %9 offset:3072 sc0 sc1"
               : "=&v"(b[0]), "=&v"(b[1]), "=&v"(b[2]), "=&v"(b[3]), "=&v"(b[4]), "=&v"(b[5]), "=&v"(b[6]), "=&v"(b[7])
               : "v"(v), "s"(s) : "memory");
}
__device__ __forceinline__ void store_rows(u32x4 a, u32x4 b, u32x4 c, u32x4 d, unsigned v, char* s) {
  asm volatile("global_store_dwordx4 %4, %0, %5 offset:-4096 sc0 sc1\n\t"
               "global_store_dwordx4 %4, %1, %5 offset:-2048 sc0 sc1\n\t"
               "global_store_dwordx4 %4, %2, %5 sc0 sc1\n\t"
               "global_store_dwordx4 %4, %3, %5 offset:2048 sc0 sc1\n\ts_nop 1"
               :: "v"(a), "v"(b), "v"(c), "v"(d), "v"(v), "s"(s) : "memory");
}

#define PIECE(b) "+v"(b[0]), "+v"(b[1]), "+v"(b[2]), "+v"(b[3]), "+v"(b[4]), "+v"(b[5]), "+v"(b[6]), "+v"(b[7])

__device__ __forceinline__ void copy_stream(char* ws, char* out, char* oflag, int c, int rank, unsigned call,
                                            LDS_AS unsigned* ho, int v, int lane, unsigned long long* dbg) {
  typedef unsigned u32x2 __attribute__((ext_vector_type(2)));
  const char* stg = ws + OFF_DATA + (size_t)(call & 1) * PARITY_B + D_STG + (size_t)c * MIB;
  const char* fp = ws + OFF_SFLAG + c * 512;
  const char* tick = ws + OFF_TICK + (7 + c) * 256;
  char* tmo = ws + OFF_TMO;
  const unsigned vl = 4096 + lane * 16;
  const unsigned vo = (unsigned)(4 * (lane >> 4) + 2) * (SHARD * 4) + (lane & 15) * 16;
  const unsigned base = (call - 1) * (NSTG - NWV) - 2 * NWV;
  unsigned long long r0 = 0, r1 = 0;
  unsigned nanm = 0;
  auto ready = [&](unsigned e) { return (((e & 1) ? r1 : r0) >> (e >> 1)) & 1; };
  auto wait_ready = [&](unsigned e) {
    const unsigned long long t0 = __builtin_amdgcn_s_memrealtime();
    while (!ready(e)) {
      u32x2 f;
      unsigned tv;
      asm volatile("global_load_dwordx2 %0, %2, %3 sc0 sc1\n\t"
                   "global_load_dword %1, %4, %5 sc1\n\ts_waitcnt vmcnt(0)"
                   : "=&v"(f), "=&v"(tv) : "v"(lane * 8), "s"(fp), "v"(0u), "s"(tmo) : "memory");
      r0 = __ballot((int)(f[0] - call) >= 0);
      r1 = __ballot((int)(f[1] - call) >= 0);
      if (ready(e)) break;
      if (expired(tv, tmo, call, t0, lane)) { r0 = r1 = ~0ull; nanm = 0x7fc00000u; break; }
      __builtin_amdgcn_s_sleep(8);
    }
  };
  u32x4 cb[8], nb[8];
  unsigned cur = v, nxt = NWV + v;
  *(volatile LDS_AS unsigned*)ho = 0;
  BARRIER();
  wait_ready(cur);
  load_piece(cb, vl, stg + cur * 8192);
  asm volatile("s_waitcnt vmcnt(0)" : PIECE(cb) :: "memory");
  wait_ready(nxt);
  {
    const unsigned long long t0 = __builtin_amdgcn_s_memrealtime();
    while (*(volatile LDS_AS unsigned*)ho == 0 && __builtin_amdgcn_s_memrealtime() - t0 < DEADLINE_TICKS)
      __builtin_amdgcn_s_sleep(1);
  }
  STAMP(dbg, 0);
  char* d;
#pragma unroll 1
  for (;;) {
    d = out + (size_t)staged_row(cur >> 1, c, rank) * (SHARD * 4) + (cur & 1) * 256;
    for (int k = 0; k < 8; ++k) cb[k] |= nanm;
    store_rows(cb[0], cb[1], cb[2], cb[3], vo, d);
    if (nxt >= (unsigned)NSTG) break;
    const bool was = ready(nxt);
    unsigned tk, tv;
    u32x2 f;
    draw(tk, tick, 1u);
    load_piece(nb, vl, stg + nxt * 8192);
    asm volatile("global_load_dwordx2 %0, %2, %3 sc0 sc1\n\t"
                 "global_load_dword %1, %4, %5 sc1\n\ts_waitcnt vmcnt(11)"
                 : "=&v"(f), "=&v"(tv) : "v"(lane * 8), "s"(fp), "v"(0u), "s"(tmo) : "memory");
    store_rows(cb[4], cb[5], cb[6], cb[7], vo + 16 * (SHARD * 4), d);
    asm volatile("s_waitcnt vmcnt(4)" : PIECE(nb), "+v"(f), "+v"(tv), "+v"(tk) :: "memory");
    r0 |= __ballot((int)(f[0] - call) >= 0);
    r1 |= __ballot((int)(f[1] - call) >= 0);
    if (__builtin_amdgcn_readfirstlane(tv) == call) nanm = 0x7fc00000u;
    if (!was) {
      wait_ready(nxt);
      load_piece(nb, vl, stg + nxt * 8192);
      asm volatile("s_waitcnt vmcnt(0)" : PIECE(nb) :: "memory");
    }
    wait_vm<0>();
    STAMP(dbg, 1 + (cur >> 4));
    for (int k = 0; k < 8; ++k) cb[k] = nb[k];
    cur = nxt;
    nxt = __builtin_amdgcn_readfirstlane(tk) - base;
  }
  if (!BLAST_C) wait_vm<0>();
  store_rows(cb[4], cb[5], cb[6], cb[7], vo + 16 * (SHARD * 4), d);
  wait_vm<0>();
  if (lane == 0) flag_store(oflag, call);
  STAMP(dbg, 1 + (cur >> 4));
}

__device__ __forceinline__ float gelu(float y) {
  const float k1 = -2.0f * 0.7978845608028654f * 1.4426950408889634f;
  const float k3 = k1 * 0.044715f;
  float e = __builtin_amdgcn_exp2f(y * (k1 + k3 * y * y));
  return y * __builtin_amdgcn_rcpf(1.0f + e);
}

#define NS1 12
#define NS2 9
__host__ __device__ constexpr int ring_slots(int nu) { return nu == 5 ? 5 : nu == 4 ? 6 : nu == 3 ? 7 : nu == 2 ? NS2 : NS1; }

struct Tile {
  unsigned a, astep, b;
  unsigned aflag, bflag;
  int acount, abase, au;
  bool alocal, blocal;
  int nu;
  char* out;
  char* oflag;
  bool staged;
};

template <int NU>
__device__ __forceinline__ void tile_loader(unsigned lbase, const char* a, unsigned astep, const char* b, int j,
                                            int lane) {
  constexpr int SLOT = NU * 4096 + 8192;
  constexpr int NS = ring_slots(NU);
  constexpr int D = NS - 1;
  constexpr int NLD = NU + 2;
  static_assert(NS * SLOT <= LDS_RING && (D - 1) * NLD <= 63, "ring");
  const unsigned voff = lane * 16;
  a += j * (NU * 1024);
  b += j * 2048;
  unsigned slot = 0;
#pragma unroll 1
  for (int t = 0; t < NSTEP; ++t) {
    BARRIER();
    const unsigned la = lbase + slot + j * (NU * 1024);
    const unsigned lb = lbase + slot + NU * 4096 + j * 2048;
    unsigned keep;
#define AG_A(off) "global_load_lds_dwordx4 %1, %2 offset:" #off " sc0 sc1\n\t"
#define AG_HEAD "s_mov_b32 %0, m0\n\ts_mov_b32 m0, %4\n\ts_nop 0\n\t"
#define AG_TAIL "s_mov_b32 m0, %5\n\ts_nop 0\n\t" \
                "global_load_lds_dwordx4 %1, %3 sc0 sc1\n\t" \
                "global_load_lds_dwordx4 %1, %3 offset:1024 sc0 sc1\n\t" \
                "s_mov_b32 m0, %0\n\ts_waitcnt vmcnt(%6)"
#define AG_OPS : "=&s"(keep) : "v"(voff), "s"(a), "s"(b), "s"(la), "s"(lb), "i"((D - 1) * NLD), \
                 "v"(voff + 4096), "s"(la + 4096) : "memory"
    if constexpr (NU == 1) asm volatile(AG_HEAD AG_A(0) AG_TAIL AG_OPS);
    if constexpr (NU == 2) asm volatile(AG_HEAD AG_A(0) AG_A(1024) AG_TAIL AG_OPS);
    if constexpr (NU == 3) asm volatile(AG_HEAD AG_A(0) AG_A(1024) AG_A(2048) AG_TAIL AG_OPS);
    if constexpr (NU == 4) asm volatile(AG_HEAD AG_A(0) AG_A(1024) AG_A(2048) AG_A(3072) AG_TAIL AG_OPS);
    if constexpr (NU == 5)
      asm volatile(AG_HEAD AG_A(0) AG_A(1024) AG_A(2048) AG_A(3072)
                   "s_mov_b32 m0, %8\n\ts_nop 0\n\tglobal_load_lds_dwordx4 %7, %2 sc0 sc1\n\t" AG_TAIL AG_OPS);
    a += astep;
    b += 16384;
    slot += SLOT;
    if (slot == NS * SLOT) slot = 0;
  }
  static_for<D>([&](auto tc) {
    constexpr int t = decltype(tc)::value;
    BARRIER();
    wait_vm<(D - 2 - t > 0 ? D - 2 - t : 0) * NLD>();
  });
  BARRIER();
}

__device__ __forceinline__ void gemm_tile(LDS_AS char* smem, char* ws, const Tile& T, unsigned call,
                                          int wave, int lane, unsigned long long* dbg) {
  const int nu = T.nu;
  const int D = ring_slots(nu) - 1;
  LDS_AS unsigned* bad = (LDS_AS unsigned*)(smem + LDS_RING);

  if (wave == 9) {
    const bool lo = lane < 32;
    const int cnt = lo ? T.acount : (T.blocal ? 16 : NWV);
    const int g = (lane & 31) < cnt ? (lane & 31) : cnt - 1;
    const bool loc = lo ? T.alocal : T.blocal;
    const char* fp = ws + (lo ? T.aflag : T.bflag) + g * FLAG_STRIDE;
    const int mul = loc ? 4096 : 2;
    const int add = (loc ? g * 16 : 0) - (lo ? T.abase : 0);
    const int per = lo ? T.au : 4;
    char* tmo = ws + OFF_TMO;
    int first = -1;
    int need = per;
    bool dead = false;
    STAMP(dbg, 0);
    for (int t = 0; t < NSTEP; ++t) {
      if (!dead && !__all(first >= need)) {
        const unsigned long long t0 = __builtin_amdgcn_s_memrealtime();
        for (;;) {
          unsigned f, tv;
          poll_once(f, tv, fp, tmo);
          const int d = (int)(f - call * FSEQ);
          first = (d > 0 ? d : 0) * mul + add;
          if (__all(first >= need)) break;
          if (expired(tv, tmo, call, t0, lane)) { dead = true; break; }
          __builtin_amdgcn_s_sleep(POLL_SLEEP);
        }
      }
      need += per;
      BARRIER();
    }
    {
      const unsigned tv = __hip_atomic_load((GLOBAL_AS unsigned*)tmo, __ATOMIC_RELAXED, __HIP_MEMORY_SCOPE_AGENT);
      if (lane == 0) *bad = (dead || tv == call) ? 0x7fc00000u : 0u;
    }
    for (int t = 0; t <= D; ++t) BARRIER();
    STAMP(dbg, 4);
  } else if (wave >= 5) {
    const char* data = ws + OFF_DATA + (size_t)(call & 1) * PARITY_B;
    const unsigned lbase = (unsigned)(uintptr_t)smem;
    const char* a = data + T.a;
    const char* b = data + T.b;
    const int j = wave - 5;
    if (nu == 4) tile_loader<4>(lbase, a, T.astep, b, j, lane);
    else if (nu == 2) tile_loader<2>(lbase, a, T.astep, b, j, lane);
    else if (nu == 1) tile_loader<1>(lbase, a, T.astep, b, j, lane);
    else if (nu == 3) tile_loader<3>(lbase, a, T.astep, b, j, lane);
    else tile_loader<5>(lbase, a, T.astep, b, j, lane);
  } else if (wave >= nu) {
    for (int t = 0; t < NSTEP + D + 1; ++t) BARRIER();
  } else {
    f32x4 acc[2][4];
    for (int m = 0; m < 2; ++m)
      for (int n = 0; n < 4; ++n) acc[m][n] = f32x4{0.f, 0.f, 0.f, 0.f};
    f16x8 fa[2][2][2], fb[2][4][2];
    LDS_AS char* pa = smem + wave * 4096 + lane * 16;
    LDS_AS char* pb = smem + nu * 4096 + lane * 16;
    const unsigned sstep = nu * 4096 + 8192, send = sstep * (D + 1);
    unsigned slot = 0;
    auto rd = [&](auto sc) {
      constexpr int s = decltype(sc)::value;
      for (int u = 0; u < 2; ++u) {
        for (int m = 0; m < 2; ++m) fa[s][m][u] = *(LDS_AS f16x8*)(pa + slot + m * 2048 + u * 1024);
        for (int n = 0; n < 4; ++n) fb[s][n][u] = *(LDS_AS f16x8*)(pb + slot + u * 4096 + n * 1024);
      }
      slot += sstep;
      if (slot == send) slot = 0;
    };
    auto mm = [&](auto sc) {
      constexpr int s = decltype(sc)::value;
      for (int u = 0; u < 2; ++u)
        for (int m = 0; m < 2; ++m)
          for (int n = 0; n < 4; ++n)
            acc[m][n] = __builtin_amdgcn_mfma_f32_16x16x32_f16(fa[s][m][u], fb[s][n][u], acc[m][n], 0, 0, 0);
    };
    using I0 = std::integral_constant<int, 0>;
    using I1 = std::integral_constant<int, 1>;
    for (int t = 0; t <= D; ++t) BARRIER();
    rd(I0{});
    for (int s = 1; s < NSTEP - 1; s += 2) {
      BARRIER();
      rd(I1{});
      __builtin_amdgcn_sched_barrier(0);
      mm(I0{});
      __builtin_amdgcn_sched_barrier(0);
      BARRIER();
      rd(I0{});
      __builtin_amdgcn_sched_barrier(0);
      mm(I1{});
      __builtin_amdgcn_sched_barrier(0);
    }
    BARRIER();
    rd(I1{});
    const unsigned isbad = *bad;
    __builtin_amdgcn_sched_barrier(0);
    mm(I0{});
    __builtin_amdgcn_sched_barrier(0);
    BARRIER();
    mm(I1{});
    char* out = T.out + (size_t)wave * (T.staged ? 2 * 8192 : 32 * SHARD * 4);
    const unsigned vo = T.staged ? lane * 16 : (unsigned)(4 * (lane >> 4)) * (SHARD * 4) + (lane & 15) * 16;
    const unsigned sj = T.staged ? 1024 : SHARD * 4, sm = T.staged ? 4096 : 16 * SHARD * 4;
    for (int m = 0; m < 2; ++m) {
      for (int jj = 0; jj < 4; ++jj) {
        u32x4 o;
        for (int n = 0; n < 4; ++n)
          o[n] = isbad | __builtin_bit_cast(unsigned, gelu(acc[m][n][jj]));
        asm volatile("global_store_dwordx4 %1, %0, %2 sc0 sc1\n\ts_nop 1"
                     :: "v"(o), "v"(vo + m * sm + jj * sj), "s"(out) : "memory");
      }
    }
    if (T.oflag != nullptr) {
      wait_vm<0>();
      if (lane == 0) flag_store(T.oflag + wave * (T.staged ? 8 : 256), call);
    }
    if (wave == 0) STAMP(dbg, 5);
  }
}

__device__ __forceinline__ Tile make_tile(const Params& P, int src, int u0, int nu, bool foreign, int xcd,
                                          unsigned call) {
  const int grp = P.rank >> 2, pos = P.rank & 3;
  const int c = xcd >> 1, ch = xcd & 1, cr = (4 - c) & 3;
  Tile T;
  int row0, R;
  bool direct = false;
  if (src < 4) {
    const int ub = src < 3 ? 256 * src : 512 + 64 * U2;
    T.au = src < 2 ? 4 : src == 2 ? U2 : UX;
    T.a = D_OWNX + ub * 4096;
    T.aflag = OFF_FLAG + (ub >> 4) * FLAG_STRIDE; T.acount = 4 * T.au; T.abase = 0; T.alocal = true;
    row0 = P.rank * SHARD + (ub >> 1);
    R = ub >> 6;
  } else if (src < 13) {
    const int i = (src - 4) / 3, b = src - 4 - 3 * i;
    T.au = b < 2 ? 4 : U2;
    T.a = D_INTRA + (3 * i + b) * MIB;
    T.aflag = OFF_FLAG + (8 + i) * 4096; T.acount = NWV; T.abase = 256 * b; T.alocal = false;
    row0 = (grp * 4 + ((pos + i + 1) & 3)) * SHARD + b * 128;
    R = b < 2 ? 8 + U2 + 12 * b + 4 * i : U2 * i;
    direct = b == 2;
  } else {
    const int k = src - 13;
    T.au = UX;
    T.a = D_CROSS + (3 * k + 1) * MIB;
    T.aflag = OFF_FLAG + (12 + k) * 4096; T.acount = NWV; T.abase = 256; T.alocal = false;
    row0 = ((grp ^ 1) * 4 + ((pos + k) & 3)) * SHARD + ROWX;
    R = 32 + U2 + ((k - c - 1) & 3) * UX;
  }
  T.nu = nu == 0 ? T.au : nu;
  T.astep = T.au * 4096;
  T.a += u0 * 4096;
  row0 += u0 * 32;
  R += u0;
  T.staged = foreign && !direct;
  T.blocal = !foreign;
  if (!foreign) {
    T.b = D_OWNW + c * MIB + ch * 8192;
    T.bflag = OFF_FLAG + 4 * 4096 + c * 16 * FLAG_STRIDE;
    T.out = (char*)P.out[P.rank & 7] + (size_t)row0 * (SHARD * 4) + xcd * 256;
    T.oflag = nullptr;
  } else {
    T.b = D_CROSS + 3 * c * MIB + ch * 8192;
    T.bflag = OFF_FLAG + (12 + c) * 4096;
    if (direct) {
      const int drank = ((grp ^ 1) * 4 + ((pos + c) & 3)) & 7;
      T.out = (char*)P.out[drank] + (size_t)row0 * (SHARD * 4) + cr * 512 + ch * 256;
      T.oflag = P.ws[drank] + OFF_OFLAG + (size_t)(cr * 64 + NWV + 2 * R + ch) * 128;
    } else {
      T.out = P.self + OFF_DATA + (size_t)(call & 1) * PARITY_B + D_STG + c * MIB + (2 * R + ch) * 8192;
      T.oflag = P.self + OFF_SFLAG + (size_t)(c * 128 + 2 * R + ch) * 4;
    }
  }
  return T;
}

__global__ void __launch_bounds__(NTHREADS, 1) ag_gemm_kernel(Params P) {
  __shared__ __attribute__((aligned(16))) char smem_raw[LDS_BYTES];
  LDS_AS char* smem = (LDS_AS char*)smem_raw;
  const int lane = threadIdx.x & 63;
  const int wave = __builtin_amdgcn_readfirstlane(threadIdx.x >> 6);
  const int bx = blockIdx.x, xcd = bx & 7, j = bx >> 3;
  const int grp = P.rank >> 2, pos = P.rank & 3;
  char* ws = P.self;
  unsigned* ctr = (unsigned*)(ws + OFF_CTR + (size_t)bx * 128);
  unsigned long long* dbg = (unsigned long long*)(ws + OFF_DBG + (size_t)bx * 512);
  if (wave == 9) STAMP(dbg, 0);

  const int link = j;
  const int sub = (xcd + j) & 7;
  if (j < JP && wave < PW) {
    const int v = sub * PW + wave;
    const char* tick = ws + OFF_TICK + link * 256;
    LDS_AS unsigned* ho = (LDS_AS unsigned*)smem + wave;
    if (link < 3) {
      const int i = link + 1, ir = 4 - i;
      char* pw = P.ws[(grp * 4 + ((pos + i) & 3)) & 7];
      push_stream<false>(nullptr, (const char*)P.x, pw, D_INTRA + (size_t)(ir - 1) * 3 * MIB, ctr, tick,
                         pw + OFF_FLAG + (7 + ir) * 4096 + v * FLAG_STRIDE, ho, v, lane, dbg + (wave & 1) * 16);
    } else {
      const int c = link - 3, cr = (4 - c) & 3;
      char* pw = P.ws[((grp ^ 1) * 4 + ((pos + c) & 3)) & 7];
      push_stream<true>((const char*)P.w + c * 512, (const char*)P.x, pw, D_CROSS + (size_t)cr * 3 * MIB, ctr, tick,
                        pw + OFF_FLAG + (12 + cr) * 4096 + v * FLAG_STRIDE, ho, v, lane, dbg + (wave & 1) * 16);
    }
    return;
  }
  if (j < JP && wave < 9 && (link < 3 || wave >= 2 * PW)) return;
  const unsigned call = *(const CONST_AS unsigned*)(uintptr_t)ctr + 1u;
  asm volatile("" :: "s"(call));
  if (j < JP) {
    const int c = link - 3, cr = (4 - c) & 3;
    if (wave < 9) {
      const int v = sub * PW + wave - PW;
      const int drank = ((grp ^ 1) * 4 + ((pos + c) & 3)) & 7;
      copy_stream(ws, (char*)P.out[drank] + cr * 512, P.ws[drank] + OFF_OFLAG + (size_t)(cr * 64 + v) * 128, c, P.rank,
                  call, (LDS_AS unsigned*)smem + (wave - PW), v, lane, dbg + 32 + (wave & 1) * 12);
      return;
    }
    BARRIER();
    if (link >= 3 && sub == 0) {
      const char* fp = ws + OFF_OFLAG + (size_t)(c * 64 + (lane < NWV + NDIR ? lane : 0)) * 128;
      char* tmo = ws + OFF_TMO;
      const unsigned long long t0 = __builtin_amdgcn_s_memrealtime();
      while (*(volatile LDS_AS unsigned*)smem == 0 && __builtin_amdgcn_s_memrealtime() - t0 < DEADLINE_TICKS)
        __builtin_amdgcn_s_sleep(32);
      for (;;) {
        unsigned f, tv;
        poll_once(f, tv, fp, tmo);
        if (__all((int)(f - call) >= 0)) break;
        if (expired(tv, tmo, call, t0, lane)) {
          if (lane == 0) flag_store(P.out[P.rank & 7] + (size_t)P.rank * SHARD * SHARD, 0x7fc00000u);
          break;
        }
        __builtin_amdgcn_s_sleep(2);
      }
      STAMP(dbg, 62);
    }
  } else {
    const int r = j - JP;
    int s0, t2 = -1;
    bool f0 = false;
    if (r < 3) { s0 = r; f0 = true; }
    else if (r < 9) { s0 = 4 + 3 * ((r - 3) >> 1); f0 = r & 1; }
    else if (r < 15) { s0 = 5 + 3 * ((r - 9) >> 1); f0 = r & 1; }
    else if (r < 19) { s0 = 13 + r - 15; }
    else if (r < 22) { s0 = 13 + (((xcd >> 1) + 1 + r - 19) & 3); f0 = true; }
    else { s0 = r - 22; }
    if (r < 9) t2 = r;
    else if (r >= 22) t2 = r - 13;
    if (r >= 9 && r < 25) {
      const int q = (r - 9) * 8 + xcd;
      const bool isw = q >= 64;
      convert_local(isw ? (const char*)P.w + ((q - 64) >> 4) * 512 : (const char*)P.x, isw, q * 16,
                    ws + OFF_DATA + (size_t)(call & 1) * PARITY_B,
                    ws + OFF_FLAG + (isw ? 4 * 4096 + (q - 64) * FLAG_STRIDE : q * FLAG_STRIDE), call, wave, lane, dbg + 40);
    }
#pragma unroll 1
    for (int n = 0; n < 3; ++n) {
      int src = s0, u0 = 0, nu = 0;
      bool foreign = n == 0 && f0;
      if (n == 1) {
        u0 = 2 * (24 - r);
        src = r >= 22 && u0 < UX ? 3 : -1;
        nu = u0 + 2 <= UX ? 2 : 1;
      }
      if (n == 2) {
        src = t2 >= 0 ? 6 + 3 * (t2 >> 2) : -1;
        u0 = t2 & 2;
        nu = u0 + 2 <= U2 ? 2 : 1;
        foreign = !(t2 & 1);
      }
      if (src < 0) continue;
      const Tile T = make_tile(P, src, u0, nu, foreign, xcd, call);
      gemm_tile(smem, ws, T, call, wave, lane, dbg + 8 + 8 * n);
    }
  }
  if (wave == 9) {
    if (lane == 0) __hip_atomic_store((GLOBAL_AS unsigned*)ctr, call, __ATOMIC_RELAXED, __HIP_MEMORY_SCOPE_AGENT);
    STAMP(dbg, 63);
  }
}

extern "C" void kernel_launch(int rank, int nranks, ncclComm_t comm,
                              hipStream_t stream,
                              void* const* d_in, const int* in_sizes, int n_in,
                              void* d_out, int out_size,
                              const int* in_dtypes, int out_dtype,
                              void* const* d_out_all,
                              void* const* d_ws_all, size_t ws_bytes) {
  (void)comm; (void)in_dtypes; (void)out_dtype; (void)d_out;
  if (nranks != NRANK || (unsigned)rank >= (unsigned)NRANK || n_in < 2 || ws_bytes < WS_NEED ||
      in_sizes[0] != SHARD * KDIM || in_sizes[1] != KDIM * SHARD || out_size != NRANK * SHARD * SHARD) {
    fprintf(stderr, "unexpected problem shape\n");
    return;
  }
  Params P;
  P.x = (const float*)d_in[0];
  P.w = (const float*)d_in[1];
  for (int p = 0; p < NRANK; ++p) {
    P.ws[p] = (char*)d_ws_all[p];
    P.out[p] = (float*)d_out_all[p];
  }
  P.self = (char*)d_ws_all[rank];
  P.rank = rank;
  P.pad = 0;
  ag_gemm_kernel<<<NWG, NTHREADS, 0, stream>>>(P);
  HIP_CHECK(hipGetLastError());
}
```

```cpp
#include <hip/hip_runtime.h>
#include <rccl/rccl.h>
#include <cstdio>
#include <cstdint>
#include <utility>

#define HIP_CHECK(expr) do { hipError_t _e = (expr); if (_e != hipSuccess) { \
    fprintf(stderr, "HIP error %s at %s:%d\n", hipGetErrorString(_e), __FILE__, __LINE__); \
    return; } } while (0)

typedef unsigned u32x4 __attribute__((ext_vector_type(4)));
typedef unsigned u32x2 __attribute__((ext_vector_type(2)));
typedef float f32x4 __attribute__((ext_vector_type(4)));
typedef _Float16 f16x8 __attribute__((ext_vector_type(8)));
#define LDS_AS __attribute__((address_space(3)))
#define CONST_AS __attribute__((address_space(4)))
#define GLOBAL_AS __attribute__((address_space(1)))

#define U2 3
#define PW 2
#define NH 4
#define S1P 2
#define S1R 0
#define RT1 16
#define RTX 16
#define POLL_SLEEP 4
#define TOP_LANES 8
#define UNP_SLEEP 4
#define FORCE_RAW 0
#define CONV_D0 100
#define CONV_DL 0
#define SND_SLEEP 4
#define AHEAD 64
#define LATE 1000
#define UNP 7
#define CONV_NU 300

constexpr int NRANK = 8;
constexpr int KDIM = 4096;
constexpr int SHARD = 512;
constexpr int UX = 8 - U2;
constexpr int ROWX = 256 + 32 * U2;
constexpr int NWV = 8 * PW;
constexpr int NHEAD = NH * NWV;
constexpr int JP = 7;
constexpr int NWG = 256;
constexpr int NTHREADS = 640;
constexpr unsigned MIB = 1u << 20;
constexpr int NSTEP = 64;
constexpr int NSTG = 2 * (32 + U2 + 3 * UX);
constexpr int NDIR = 2 * 3 * U2;

constexpr size_t OFF_CTR = 0;
constexpr size_t OFF_TMO = 32768;
constexpr size_t OFF_TICK = 36864;
constexpr size_t OFF_OFLAG = 131072;
constexpr size_t OFF_DBG = 262144;
constexpr size_t OFF_DONE = 786432;
constexpr size_t OFF_DATA = MIB;
constexpr size_t PARITY_B = 62 * MIB;
constexpr size_t D_OWNX = 0, D_OWNW = 4 * MIB, D_INTRA = 8 * MIB, D_CROSS = 17 * MIB, D_STG = 29 * MIB;
constexpr size_t PK_OFF = 25 * MIB;
constexpr size_t D_IMG = 54 * MIB;
constexpr unsigned PK_FLAG = 7680;
constexpr int NUW = 8 * UNP;
constexpr size_t WS_NEED = OFF_DATA + 2 * PARITY_B;
static_assert(D_CROSS == D_INTRA + 9 * MIB && D_INTRA + PK_OFF >= D_STG + 4 * MIB && D_STG + PK_OFF <= D_IMG &&
              D_IMG + 8 * MIB <= PARITY_B && NSTG <= 128 && NWV + NDIR <= 64 && U2 == 3 && UNP <= 7, "layout");

constexpr unsigned long long DEADLINE_TICKS = 40000000ull;

constexpr int LDS_RING = 147456;
constexpr int LDS_BYTES = LDS_RING + 64;

struct Params {
  const float* x;
  const float* w;
  char* self;
  char* ws[NRANK];
  float* out[NRANK];
  int rank;
  int pad;
};

template <int... I, class F>
__device__ __forceinline__ void static_for_impl(std::integer_sequence<int, I...>, F&& f) {
  (f(std::integral_constant<int, I>{}), ...);
}
template <int N, class F>
__device__ __forceinline__ void static_for(F&& f) {
  static_for_impl(std::make_integer_sequence<int, N>{}, f);
}

#define BARRIER() do { asm volatile("" ::: "memory"); __builtin_amdgcn_s_waitcnt(0xC07F); \
    asm volatile("s_barrier" ::: "memory"); } while (0)

#define STAMP(dbg, id) do { } while (0)

__device__ __forceinline__ unsigned pk16(float a, float b) {
  unsigned r;
  asm("v_cvt_pk_f16_f32 %0, %1, %2" : "=v"(r) : "v"(a), "v"(b));
  return r;
}

__device__ __forceinline__ bool first_lane(int lane) {
  asm volatile("" : "+v"(lane));
  return lane == 0;
}
__device__ __forceinline__ char* uniform(char* p) {
  const unsigned long long a = (unsigned long long)p;
  unsigned lo = __builtin_amdgcn_readfirstlane((unsigned)a), hi = __builtin_amdgcn_readfirstlane((unsigned)(a >> 32));
  asm volatile("s_nop 4" : "+s"(lo), "+s"(hi));
  return (char*)((unsigned long long)hi << 32 | lo);
}
__device__ __forceinline__ void flag_store(void* p, unsigned v) {
  __hip_atomic_store((GLOBAL_AS unsigned*)p, v, __ATOMIC_RELAXED, __HIP_MEMORY_SCOPE_SYSTEM);
}
__device__ __forceinline__ bool expired(char* tmo, unsigned call, unsigned long long t0, int lane) {
  const unsigned long long dt = __builtin_amdgcn_s_memrealtime() - t0;
  if (dt < 10000) return false;
  const unsigned tv = __hip_atomic_load((GLOBAL_AS unsigned*)tmo, __ATOMIC_RELAXED, __HIP_MEMORY_SCOPE_AGENT);
  if (__builtin_amdgcn_readfirstlane(tv) == call || dt > DEADLINE_TICKS) {
    if (first_lane(lane)) __hip_atomic_store((GLOBAL_AS unsigned*)tmo, call, __ATOMIC_RELAXED, __HIP_MEMORY_SCOPE_AGENT);
    return true;
  }
  return false;
}

__device__ __forceinline__ void load_unit(f32x4 (&b)[8], const unsigned (&v)[8], const char* s) {
  asm volatile("global_load_dwordx4 %0, %8, %16\n\t"
               "global_load_dwordx4 %1, %9, %16\n\t"
               "global_load_dwordx4 %2, %10, %16\n\t"
               "global_load_dwordx4 %3, %11, %16\n\t"
               "global_load_dwordx4 %4, %12, %16\n\t"
               "global_load_dwordx4 %5, %13, %16\n\t"
               "global_load_dwordx4 %6, %14, %16\n\t"
               "global_load_dwordx4 %7, %15, %16"
               : "=&v"(b[0]), "=&v"(b[1]), "=&v"(b[2]), "=&v"(b[3]), "=&v"(b[4]), "=&v"(b[5]), "=&v"(b[6]), "=&v"(b[7])
               : "v"(v[0]), "v"(v[1]), "v"(v[2]), "v"(v[3]), "v"(v[4]), "v"(v[5]), "v"(v[6]), "v"(v[7]), "s"(s)
               : "memory");
}
template <int CNT>
__device__ __forceinline__ void wait_unit(f32x4 (&b)[8]) {
  asm volatile("s_waitcnt vmcnt(%8)"
               : "+v"(b[0]), "+v"(b[1]), "+v"(b[2]), "+v"(b[3]), "+v"(b[4]), "+v"(b[5]), "+v"(b[6]), "+v"(b[7])
               : "i"(CNT) : "memory");
}
template <int CNT>
__device__ __forceinline__ void wait_vm() {
  asm volatile("s_waitcnt vmcnt(%0)" :: "i"(CNT) : "memory");
}
__device__ __forceinline__ void cvt_unit(unsigned* d, const f32x4 (&b)[8], bool isw) {
  if (isw) {
    for (int n = 0; n < 4; ++n)
      for (int e = 0; e < 4; ++e) d[4 * n + e] = pk16(b[2 * e][n], b[2 * e + 1][n]);
  } else {
    for (int u = 0; u < 8; ++u) {
      d[2 * u] = pk16(b[u][0], b[u][1]);
      d[2 * u + 1] = pk16(b[u][2], b[u][3]);
    }
  }
}
__device__ __forceinline__ void store_4k(const u32x4 (&d)[4], unsigned v, char* p, unsigned long long tail = ~0ull) {
  asm volatile("global_store_dwordx4 %4, %0, %5 sc0 sc1\n\t"
               "global_store_dwordx4 %4, %1, %5 offset:1024 sc0 sc1\n\t"
               "global_store_dwordx4 %4, %2, %5 offset:2048 sc0 sc1\n\t"
               "s_mov_b64 exec, %6\n\t"
               "global_store_dwordx4 %4, %3, %5 offset:3072 sc0 sc1\n\t"
               "s_mov_b64 exec, -1\n\ts_nop 0"
               :: "v"(d[0]), "v"(d[1]), "v"(d[2]), "v"(d[3]), "v"(v), "s"(p), "s"(tail) : "memory");
}
__device__ __forceinline__ void gather(u32x4 (&q)[4], const unsigned* d) {
  for (int k = 0; k < 4; ++k) q[k] = u32x4{d[4 * k], d[4 * k + 1], d[4 * k + 2], d[4 * k + 3]};
}
__device__ __forceinline__ void store_4k(const unsigned* d, unsigned v, char* p, unsigned long long tail = ~0ull) {
  u32x4 q[4];
  gather(q, d);
  store_4k(q, v, p, tail);
}
__device__ __forceinline__ void unit_offsets(unsigned (&v)[8], bool isw, int lane) {
  const int q = lane >> 4, c = lane & 15;
  for (int j = 0; j < 8; ++j)
    v[j] = isw ? (unsigned)((j >> 2) * 16 + 4 * q + (j & 3)) * (SHARD * 4) + c * 16
               : (unsigned)((j >> 2) * 16 + c) * (KDIM * 4) + (j & 3) * 64 + q * 16;
}
__device__ __forceinline__ unsigned w_unit(unsigned n) {
  return ((n >> 2) * 2 + (n & 1)) * (32 * SHARD * 4) + ((n >> 1) & 1) * 256;
}
__device__ __forceinline__ unsigned x_unit(unsigned n) {
  unsigned step, un;
  if (n < 512) { step = (n & 255) >> 2; un = (n >> 8) * 4 + (n & 3); }
  else if (n < 512 + 64 * U2) { step = (n - 512) / U2; un = 8 + (n - 512) % U2; }
  else { step = (n - 512 - 64 * U2) / UX; un = 8 + U2 + (n - 512 - 64 * U2) % UX; }
  return un * (32 * KDIM * 4) + step * 256;
}

__device__ __forceinline__ unsigned w_flag(unsigned c) { return (c & 1) * 64 + (c >> 1); }
__device__ __forceinline__ unsigned flag_at(unsigned block, unsigned i) { return block * 2048 + (i & 15) * 128 + (i >> 4) * 4; }
__device__ __forceinline__ unsigned stream_flag(bool cross, unsigned c) {
  return !cross ? flag_at(c >> 7, c & 127) : c < 128 ? flag_at(0, w_flag(c)) : flag_at(1, c - 128);
}

__device__ __forceinline__ void look(int& first, bool on, const char* fp, unsigned call, int at) {
  u32x4 f;
  on = on && first != 4096;
  asm volatile("s_mov_b64 exec, %2\n\t"
               "global_load_dwordx4 %0, %1, off sc0 sc1\n\t"
               "s_mov_b64 exec, -1\n\ts_waitcnt vmcnt(0)"
               : "=&v"(f) : "v"(fp), "s"(__ballot(on)) : "memory");
  int now = 4096;
  for (int k = 3; k >= 0; --k) now = (int)(f[k] - call) >= 0 ? now : at + 16 * k;
  first = on ? now : first;
}

__device__ __forceinline__ unsigned pk_add(unsigned a, unsigned b) {
  typedef unsigned short u16x2 __attribute__((ext_vector_type(2)));
  return __builtin_bit_cast(unsigned, (u16x2)(__builtin_bit_cast(u16x2, a) + __builtin_bit_cast(u16x2, b)));
}
__device__ __forceinline__ unsigned bfi(unsigned mask, unsigned a, unsigned b) { return (a & mask) | (b & ~mask); }
constexpr unsigned HOLE = 0x60006000u;

__device__ __forceinline__ void encode_chunk(unsigned (&m)[32], unsigned (&x)[32], float mx,
                                                 LDS_AS unsigned short* list, int lane) {
  const unsigned le = pk16(mx, mx) >> 26;
  unsigned top = 0;
  for (unsigned bit = 16; bit; bit >>= 1) {
    unsigned above = __popcll(__ballot(le >= top + bit));
    asm("" : "+s"(above));
    if (above > TOP_LANES) top += bit;
  }
  top = top < 7 ? 7 : top;
  asm("" : "+s"(top));
  const unsigned base = top - 7;
  const unsigned down = ((64 - base) & 63) * 0x04000400u;
  unsigned P[4];
  for (int g = 0; g < 4; ++g) {
    for (int i = 0; i < 8; ++i) x[8 * g + i] = pk_add(x[8 * g + i], down);
    P[g] = x[8 * g + 7] << 1;
    for (int i = 0; i < 7; ++i) P[g] = bfi(0x00030003u << (2 * i), x[8 * g + i] >> (13 - 2 * i), P[g]);
  }
  unsigned n = 0;
#pragma unroll 1
  do {
    for (int g = 0; g < 4; ++g) {
      const unsigned p = P[g];
      const unsigned long long mask = __ballot(p != 0);
      const unsigned sh = __builtin_ctz(p | 0x80000000u) & 30;
      const unsigned at = __builtin_amdgcn_mbcnt_hi((unsigned)(mask >> 32), __builtin_amdgcn_mbcnt_lo((unsigned)mask, 0));
      *(volatile LDS_AS unsigned short*)(list + n + (p != 0 ? at : 64)) = (lane * 4 + g) << 6 | sh << 1 | ((p >> sh) & 3);
      P[g] = p & ~(3u << sh);
      n += __popcll(mask);
    }
  } while (__ballot((P[0] | P[1] | P[2] | P[3]) != 0) != 0 && n <= 128);
  const u32x4 esc = *(volatile LDS_AS u32x4*)(list + 8 * (lane & 15));
  for (int g = 0; g < 4; ++g) {
    const unsigned y = bfi(0x20002000u, x[8 * g + 7] >> 2, x[8 * g + 7]);
    for (int i = 0; i < 7; ++i) m[7 * g + i] = bfi(HOLE, y << (13 - 2 * i), x[8 * g + i]);
    m[28 + g] = esc[g];
  }
  if (lane >= 16) m[28] = FORCE_RAW || n > 128 ? 32 : base | n << 8;
}

__device__ __forceinline__ void decode_half(unsigned (&d)[16], const unsigned* m, unsigned p0, unsigned p1,
                                            unsigned base) {
  const unsigned up = base * 0x04000400u;
  for (int g = 0; g < 2; ++g) {
    const unsigned p = g ? p1 : p0;
    unsigned y = m[7 * g] >> 13;
    for (int i = 0; i < 7; ++i) {
      if (i) y = bfi(0x00030003u << (2 * i), m[7 * g + i] >> (13 - 2 * i), y);
      d[8 * g + i] = pk_add(bfi(HOLE, p << (13 - 2 * i), m[7 * g + i]), up);
    }
    d[8 * g + 7] = pk_add(bfi(HOLE, p >> 1, bfi(0x80008000u, y << 2, y)), up);
  }
}

__device__ __forceinline__ int staged_row(int R, int c, int rank) {
  const int grp = rank >> 2, pos = rank & 3;
  if (R < 8 + U2) return rank * SHARD + R * 32;
  R -= 8 + U2;
  if (R < 24) {
    const int b = R >= 12 ? 1 : 0, m = R - 12 * b;
    return (grp * 4 + ((pos + 1 + (m >> 2)) & 3)) * SHARD + b * 128 + (m & 3) * 32;
  }
  R -= 24;
  const int k = R / UX;
  return ((grp ^ 1) * 4 + ((pos + c + 1 + k) & 3)) * SHARD + ROWX + (R - k * UX) * 32;
}

__device__ __forceinline__ void store_rows(const u32x4 (&d)[4], unsigned v, char* s) {
  asm volatile("global_store_dwordx4 %4, %0, %5 offset:-4096 sc0 sc1\n\t"
               "global_store_dwordx4 %4, %1, %5 offset:-2048 sc0 sc1\n\t"
               "global_store_dwordx4 %4, %2, %5 sc0 sc1\n\t"
               "global_store_dwordx4 %4, %3, %5 offset:2048 sc0 sc1\n\ts_nop 1"
               :: "v"(d[0]), "v"(d[1]), "v"(d[2]), "v"(d[3]), "v"(v), "s"(s) : "memory");
}
__device__ __forceinline__ void load_4k(u32x4 (&d)[4], unsigned v, const char* s) {
  asm volatile("global_load_dwordx4 %0, %4, %5 sc0 sc1\n\t"
               "global_load_dwordx4 %1, %4, %5 offset:1024 sc0 sc1\n\t"
               "global_load_dwordx4 %2, %4, %5 offset:2048 sc0 sc1\n\t"
               "global_load_dwordx4 %3, %4, %5 offset:3072 sc0 sc1"
               : "+v"(d[0]), "+v"(d[1]), "+v"(d[2]), "+v"(d[3]) : "v"(v), "s"(s) : "memory");
}
template <int CNT>
__device__ __forceinline__ void wait_4k(u32x4 (&d)[4]) {
  asm volatile("s_waitcnt vmcnt(%4)" : "+v"(d[0]), "+v"(d[1]), "+v"(d[2]), "+v"(d[3]) : "i"(CNT) : "memory");
}
__device__ __forceinline__ bool packed(bool cross, unsigned c) { return c >= NHEAD && c < (cross ? 32 * (4 + UX) - RTX : 256 - RT1); }

template <bool CROSS>
__device__ __forceinline__ void send_stream(const Params& P, int peer, int c, unsigned block, const unsigned* ctr,
                                            const char* tick, int v, int lane, unsigned long long* dbg) {
  constexpr unsigned NCH = 32 * (CROSS ? 4 + UX : 8 + U2);
  constexpr unsigned TOT = NCH + (CROSS ? NSTG : 0);
  const unsigned vs = lane * 16;
  u32x4 A[4], B[4];
  f32x4 b0[8], b1[8];
  unsigned voff[8], t[16];
  unit_offsets(voff, CROSS, lane);
  const char* src = CROSS ? (const char*)P.w + c * 512 : (const char*)P.x;
  __builtin_amdgcn_s_waitcnt(0x0F70);
  load_unit(b0, voff, src + (CROSS ? w_unit(2 * v) : x_unit(2 * v)));
  load_unit(b1, voff, src + (CROSS ? w_unit(2 * v + 1) : x_unit(2 * v + 1)));
  const unsigned call = *(const CONST_AS unsigned*)(uintptr_t)ctr + 1u;
  asm volatile("" :: "s"(call));
  BARRIER();
  const unsigned par = call & 1;
  const int cr = (4 - c) & 3;
  char* dst = P.ws[peer] + OFF_DATA + (size_t)par * PARITY_B + (size_t)block * MIB;
  char* done = P.ws[peer] + OFF_DONE + par * 131072 + block * 2048;
  unsigned cur = v, nxt = NWV + v;
  wait_unit<8>(b0);
  STAMP(dbg, 15);
  static_for<NH - 1>([&](auto hc) {
    cvt_unit(t, b0, CROSS);
    store_4k(t, vs, dst + cur * 8192);
    load_unit(b0, voff, src + (CROSS ? w_unit(2 * nxt) : x_unit(2 * nxt)));
    wait_unit<8>(b1);
    cvt_unit(t, b1, CROSS);
    store_4k(t, vs + 4096, dst + cur * 8192);
    load_unit(b1, voff, src + (CROSS ? w_unit(2 * nxt + 1) : x_unit(2 * nxt + 1)));
    wait_unit<8>(b0);
    STAMP(dbg, 1 + (cur >> 5));
    cur = nxt;
    nxt += NWV;
    if (decltype(hc)::value < NH - 2 && first_lane(lane)) flag_store(done + stream_flag(CROSS, cur - NWV), call);
  });
  cvt_unit(t, b0, CROSS);
  gather(A, t);
  wait_unit<0>(b1);
  if (first_lane(lane)) flag_store(done + stream_flag(CROSS, cur - NWV), call);
  cvt_unit(t, b1, CROSS);
  gather(B, t);

  const char* mine = P.self + OFF_DONE + par * 131072 + (lane & 15) * 128 + (lane >> 4) * 16;
  char* out = (char*)P.out[peer] + cr * 512;
  char* data = P.self + OFF_DATA + (size_t)par * PARITY_B;
  char* tmo = P.self + OFF_TMO;
  const unsigned vo = (unsigned)(4 * (lane >> 4) + 2) * (SHARD * 4) + (lane & 15) * 16;
  const unsigned base = (call - 1) * (TOT - NHEAD) - NHEAD - NWV;
  unsigned info = 32, seen = ~0u;
  int first = 0;
#pragma unroll 1
  for (;;) {
    const bool last = nxt >= TOT, piece = CROSS && cur >= NCH;
    const unsigned e = cur - NCH;
    char* to = piece ? out + (size_t)staged_row(e >> 1, c, P.rank) * (SHARD * 4) + (e & 1) * 256
                     : dst + cur * 8192 + (info == 32 ? 0 : PK_OFF);
    if (piece) store_rows(A, vo, to); else store_4k(A, vs, to);
    unsigned tk, inf;
    asm volatile("s_mov_b64 exec, 1\n\t"
                 "global_atomic_add %0, %1, %2, %3 sc0\n\t"
                 "s_mov_b64 exec, -1"
                 : "=&v"(tk) : "v"(0u), "v"(last ? 0u : 1u), "s"(tick) : "memory");
    unsigned from = nxt, blk = nxt >> 7, at = nxt & 127;
    if (CROSS) {
      if (nxt < 128) { from = 512 + 128 * c + nxt; blk = 4 + c; at = w_flag(nxt); }
      else if (nxt < NCH) { from = nxt + 128 + 32 * U2; blk = 3; at = nxt - 128; }
      else { blk = D_STG / MIB + c; at = nxt - NCH; from = blk * 128 + at; }
    }
    const int who = (at & 15) | (at >> 6) << 4;
    bool ok = false;
    if (!last) {
      if (blk != seen) { seen = blk; first = 0; }
      if (__builtin_amdgcn_readlane(first, who) <= (int)at)
        look(first, true, mine + blk * 2048, call, (lane & 15) + (lane >> 4) * 64);
      ok = __builtin_amdgcn_readlane(first, who) > (int)at;
    }
    bool image = packed(CROSS, nxt);
    const char* next = uniform(data + (ok ? (image ? (unsigned)D_IMG : 0u) + from * 8192 : 0u));
    load_4k(A, vs, next);
    asm volatile("global_load_dword %0, %1, %2 sc0 sc1" : "=&v"(inf) : "v"(7168u + 256u), "s"(next) : "memory");
    if (S1P != S1R && info == 32) wait_vm<6 + S1R>();
    wait_4k<6 + S1P>(B);
    if (piece) store_rows(B, vo + 16 * (SHARD * 4), to);
    else store_4k(B, vs + 4096, to, info == 32 ? ~0ull : info >= (97 << 8) ? 0xffff : info >= (65 << 8) ? 0xfff
                                                       : info >= (33 << 8) ? 0xff : 0xf);
    load_4k(B, vs + 4096, next);
    wait_4k<4>(A);
    asm volatile("" : "+v"(tk), "+v"(inf));
    if (!piece && first_lane(lane)) {
      if (info == 32) flag_store(done + stream_flag(CROSS, cur), call);
      if (packed(CROSS, cur))
        __hip_atomic_store((GLOBAL_AS unsigned long long*)(dst + cur * 8192 + PK_OFF + PK_FLAG),
                           (unsigned long long)call << 32 | info, __ATOMIC_RELAXED, __HIP_MEMORY_SCOPE_SYSTEM);
    }
    STAMP(dbg, 1 + (cur >> 5));
    if (last) break;
    cur = nxt;
    nxt = __builtin_amdgcn_readfirstlane(tk) - base;
    info = image ? __builtin_amdgcn_readfirstlane(inf) : 32;
    if (!ok || (image && info == 32)) {
      const unsigned long long t0 = __builtin_amdgcn_s_memrealtime();
      bool dead = false;
      while (__builtin_amdgcn_readlane(first, who) <= (int)at) {
        look(first, lane == who, mine + blk * 2048, call, (lane & 15) + (lane >> 4) * 64);
        if (__builtin_amdgcn_readlane(first, who) > (int)at) break;
        if (expired(tmo, call, t0, lane)) { dead = true; break; }
        __builtin_amdgcn_s_sleep(SND_SLEEP);
      }
      wait_4k<0>(B);
#pragma unroll 1
      for (image = image && !ok;; image = false) {
        const char* again = uniform(data + (image ? (unsigned)D_IMG : 0u) + from * 8192);
        load_4k(A, vs, again);
        load_4k(B, vs + 4096, again);
        wait_4k<0>(A);
        wait_4k<0>(B);
        info = image ? __builtin_amdgcn_readlane(B[3][0], 16) : 32;
        if (!image || info != 32) break;
      }
      if (dead) {
        for (int k = 0; k < 4; ++k) A[k] = B[k] = u32x4{0x7fc07fc0u, 0x7fc07fc0u, 0x7fc07fc0u, 0x7fc07fc0u};
        info = 32;
      }
    }
  }
  wait_4k<0>(B);
  if (CROSS && first_lane(lane)) flag_store(P.ws[peer] + OFF_OFLAG + (size_t)(cr * 64 + v) * 128, call);
  STAMP(dbg, 14);
}

__device__ __forceinline__ void unpack_stream(char* ws, int link, unsigned call, LDS_AS unsigned* plane, int u,
                                              int lane, unsigned long long* dbg) {
  const unsigned block = D_INTRA / MIB + link * 3;
  const unsigned end = link < 3 ? 256 - RT1 : 32 * (4 + UX) - RTX;
  char* whole = ws + OFF_DATA + (size_t)(call & 1) * PARITY_B + (size_t)block * MIB;
  char* done = ws + OFF_DONE + (call & 1) * 131072 + block * 2048;
  char* tmo = ws + OFF_TMO;
  const unsigned vs = lane * 16;
  unsigned owed = ~0u;
#pragma unroll 1
  for (unsigned cur = NHEAD + u; cur < end; cur += NUW) {
    const char* slot = whole + cur * 8192 + PK_OFF;
    *(volatile LDS_AS u32x4*)(plane + 4 * lane) = u32x4{0, 0, 0, 0};
    unsigned info = 32;
    const unsigned long long t0 = __builtin_amdgcn_s_memrealtime();
    for (;;) {
      u32x2 f;
      asm volatile("global_load_dwordx2 %0, %1, %2 sc0 sc1\n\ts_waitcnt vmcnt(0)"
                   : "=&v"(f) : "v"(PK_FLAG), "s"(slot) : "memory");
      if (owed != ~0u) {
        if (first_lane(lane)) flag_store(done + owed, call);
        owed = ~0u;
      }
      if ((int)(__builtin_amdgcn_readfirstlane(f[1]) - call) >= 0) { info = __builtin_amdgcn_readfirstlane(f[0]); break; }
      if (expired(tmo, call, t0, lane)) break;
      __builtin_amdgcn_s_sleep(UNP_SLEEP);
    }
    if (info == 32) continue;
    u32x4 q[7];
    unsigned esc;
    asm volatile("global_load_dword %7, %9, %10 offset:3072 sc0 sc1\n\t"
                 "global_load_dwordx4 %0, %8, %10 offset:-4096 sc0 sc1\n\t"
                 "global_load_dwordx4 %1, %8, %10 offset:-3072 sc0 sc1\n\t"
                 "global_load_dwordx4 %2, %8, %10 offset:-2048 sc0 sc1\n\t"
                 "global_load_dwordx4 %3, %8, %10 offset:-1024 sc0 sc1\n\t"
                 "global_load_dwordx4 %4, %8, %10 sc0 sc1\n\t"
                 "global_load_dwordx4 %5, %8, %10 offset:1024 sc0 sc1\n\t"
                 "global_load_dwordx4 %6, %8, %10 offset:2048 sc0 sc1\n\t"
                 "s_waitcnt vmcnt(0)"
                 : "=&v"(q[0]), "=&v"(q[1]), "=&v"(q[2]), "=&v"(q[3]), "=&v"(q[4]), "=&v"(q[5]), "=&v"(q[6]),
                   "=&v"(esc)
                 : "v"(vs), "v"(vs >> 2), "s"(slot + 4096) : "memory");
    const unsigned n = info >> 8;
    for (int h = 0; h < 2; ++h) {
      const unsigned e = esc >> (16 * h);
      const unsigned val = 2 * lane + h < n ? (e & 3) << ((e >> 1) & 30) : 0;
      __hip_atomic_fetch_or(plane + ((e >> 6) & 255), val, __ATOMIC_RELAXED, __HIP_MEMORY_SCOPE_WORKGROUP);
    }
    asm volatile("" ::: "memory");
    const u32x4 P = *(volatile LDS_AS u32x4*)(plane + 4 * lane);
    unsigned m[28];
    for (int k = 0; k < 28; ++k) m[k] = q[k >> 2][k & 3];
    for (int h = 0; h < 2; ++h) {
      unsigned d[16];
      decode_half(d, m + 14 * h, P[2 * h], P[2 * h + 1], info & 31);
      store_4k(d, vs, whole + cur * 8192 + 4096 * h);
    }
    owed = stream_flag(link >= 3, cur);
  }
  wait_vm<0>();
  if (owed != ~0u && first_lane(lane)) flag_store(done + owed, call);
  STAMP(dbg, 3);
}

__device__ __forceinline__ void convert_chunk(const Params& P, unsigned own, unsigned call, LDS_AS unsigned short* list,
                                              int lane) {
  const bool isw = own >= 512;
  const char* src = isw ? (const char*)P.w + ((own - 512) >> 7) * 512 : (const char*)P.x;
  char* data = P.self + OFF_DATA + (size_t)(call & 1) * PARITY_B + (size_t)own * 8192;
  f32x4 buf[2][8];
  unsigned voff[8], x[32];
  unit_offsets(voff, isw, lane);
  load_unit(buf[0], voff, src + (isw ? w_unit(2 * own & 255) : x_unit(2 * own)));
  load_unit(buf[1], voff, src + (isw ? w_unit((2 * own + 1) & 255) : x_unit(2 * own + 1)));
  wait_unit<8>(buf[0]);
  wait_unit<0>(buf[1]);
  float mx = 0.f;
  for (int h = 0; h < 2; ++h) {
    cvt_unit(x + 16 * h, buf[h], isw);
    store_4k(x + 16 * h, lane * 16, data + 4096 * h);
    for (int j = 0; j < 8; j += 2)
      asm("v_max3_f32 %0, |%1|, |%2|, %0" : "+v"(mx) : "v"(buf[h][j][0]), "v"(buf[h][j + 1][3]));
  }
  if (isw ? (own & 127) >= NHEAD : own < 256 + 32 * U2 ? packed(false, own) : packed(true, own - 128 - 32 * U2)) {
    unsigned m[32];
    encode_chunk(m, x, mx, list, lane);
    store_4k(m, lane * 16, data + D_IMG);
    store_4k(m + 16, lane * 16, data + D_IMG + 4096, 0xfffffull);
  }
  wait_vm<0>();
  const unsigned at = isw ? flag_at(own >> 7, w_flag(own & 127))
                          : own < 256 + 32 * U2 ? flag_at(own >> 7, own & 127) : flag_at(3, own - 256 - 32 * U2);
  if (first_lane(lane)) flag_store(P.self + OFF_DONE + (call & 1) * 131072 + at, call);
}

__device__ __forceinline__ float gelu(float y) {
  const float k1 = -2.0f * 0.7978845608028654f * 1.4426950408889634f;
  const float k3 = k1 * 0.044715f;
  float e = __builtin_amdgcn_exp2f(y * (k1 + k3 * y * y));
  return y * __builtin_amdgcn_rcpf(1.0f + e);
}

#define NS1 12
#define NS2 9
__host__ __device__ constexpr int ring_slots(int nu) { return nu == 5 ? 5 : nu == 4 ? 6 : nu == 3 ? 7 : nu == 2 ? NS2 : NS1; }

struct Tile {
  unsigned a, b;
  unsigned aflag, bflag;
  int au, nu;
  char* out;
  char* oflag;
  int piece;
  bool staged;
};

template <int NU>
__device__ __forceinline__ void tile_loader(unsigned lbase, const char* a, unsigned astep, const char* b, int j,
                                            int lane) {
  constexpr int SLOT = NU * 4096 + 8192;
  constexpr int NS = ring_slots(NU);
  constexpr int D = NS - 1;
  constexpr int NLD = NU + 2;
  static_assert(NS * SLOT <= LDS_RING && (D - 1) * NLD <= 63, "ring");
  const unsigned voff = lane * 16;
  a += j * (NU * 1024);
  b += j * 2048;
  unsigned slot = 0;
#pragma unroll 1
  for (int t = 0; t < NSTEP; ++t) {
    BARRIER();
    const unsigned la = lbase + slot + j * (NU * 1024);
    const unsigned lb = lbase + slot + NU * 4096 + j * 2048;
    unsigned keep;
#define AG_A(off) "global_load_lds_dwordx4 %1, %2 offset:" #off " sc0 sc1\n\t"
#define AG_HEAD "s_mov_b32 %0, m0\n\ts_mov_b32 m0, %4\n\ts_nop 0\n\t"
#define AG_TAIL "s_mov_b32 m0, %5\n\ts_nop 0\n\t" \
                "global_load_lds_dwordx4 %1, %3 sc0 sc1\n\t" \
                "global_load_lds_dwordx4 %1, %3 offset:1024 sc0 sc1\n\t" \
                "s_mov_b32 m0, %0\n\ts_waitcnt vmcnt(%6)"
#define AG_OPS : "=&s"(keep) : "v"(voff), "s"(a), "s"(b), "s"(la), "s"(lb), "i"((D - 1) * NLD), \
                 "v"(voff + 4096), "s"(la + 4096) : "memory"
    if constexpr (NU == 1) asm volatile(AG_HEAD AG_A(0) AG_TAIL AG_OPS);
    if constexpr (NU == 2) asm volatile(AG_HEAD AG_A(0) AG_A(1024) AG_TAIL AG_OPS);
    if constexpr (NU == 3) asm volatile(AG_HEAD AG_A(0) AG_A(1024) AG_A(2048) AG_TAIL AG_OPS);
    if constexpr (NU == 4) asm volatile(AG_HEAD AG_A(0) AG_A(1024) AG_A(2048) AG_A(3072) AG_TAIL AG_OPS);
    if constexpr (NU == 5)
      asm volatile(AG_HEAD AG_A(0) AG_A(1024) AG_A(2048) AG_A(3072)
                   "s_mov_b32 m0, %8\n\ts_nop 0\n\tglobal_load_lds_dwordx4 %7, %2 sc0 sc1\n\t" AG_TAIL AG_OPS);
    a += astep;
    b += 16384;
    slot += SLOT;
    if (slot == NS * SLOT) slot = 0;
  }
  static_for<D>([&](auto tc) {
    constexpr int t = decltype(tc)::value;
    BARRIER();
    wait_vm<(D - 2 - t > 0 ? D - 2 - t : 0) * NLD>();
  });
  BARRIER();
}

__device__ __forceinline__ void gemm_tile(LDS_AS char* smem, char* ws, const Tile& T, unsigned call,
                                          int wave, int lane, unsigned long long* dbg) {
  const int nu = T.nu;
  const int D = ring_slots(nu) - 1;
  LDS_AS unsigned* bad = (LDS_AS unsigned*)(smem + LDS_RING);

  if (wave == 9) {
    const bool lo = lane < 48;
    const int g = lo ? (lane & 15) + (lane >> 4) * 64 : lane - 48;
    const char* fp = ws + OFF_DONE + (call & 1) * 131072 + (lo ? T.aflag + (lane >> 4) * 16 : T.bflag) + (lane & 15) * 128;
    const int per = lo ? T.au : 2;
    char* tmo = ws + OFF_TMO;
    int first = g < 32 * per ? g : 4096;
    int need = per;
    int ahead = 4096;
    bool dead = false;
    STAMP(dbg, 0);
    for (int t = 0; t < NSTEP; ++t) {
      if (!dead && !__all(2 * first >= need)) {
        const unsigned long long t0 = __builtin_amdgcn_s_memrealtime();
        for (;;) {
          look(first, 2 * first < need + ahead, fp, call, g);
          ahead = AHEAD;
          if (__all(2 * first >= need)) break;
          if (expired(tmo, call, t0, lane)) { dead = true; break; }
          if (t == 0) __builtin_amdgcn_s_sleep(4 * POLL_SLEEP); else __builtin_amdgcn_s_sleep(POLL_SLEEP);
        }
      }
      need += per;
      BARRIER();
    }
    {
      const unsigned tv = __hip_atomic_load((GLOBAL_AS unsigned*)tmo, __ATOMIC_RELAXED, __HIP_MEMORY_SCOPE_AGENT);
      if (first_lane(lane)) *bad = (dead || tv == call) ? 0x7fc00000u : 0u;
    }
    for (int t = 0; t <= D; ++t) BARRIER();
    STAMP(dbg, 4);
  } else if (wave >= 5) {
    const char* data = ws + OFF_DATA + (size_t)(call & 1) * PARITY_B;
    const unsigned lbase = (unsigned)(uintptr_t)smem;
    const char* a = data + T.a;
    const char* b = data + T.b;
    const int j = wave - 5;
    if (nu == 4) tile_loader<4>(lbase, a, T.au * 4096, b, j, lane);
    else if (nu == 2) tile_loader<2>(lbase, a, T.au * 4096, b, j, lane);
    else if (nu == 1) tile_loader<1>(lbase, a, T.au * 4096, b, j, lane);
    else if (nu == 3) tile_loader<3>(lbase, a, T.au * 4096, b, j, lane);
    else tile_loader<5>(lbase, a, T.au * 4096, b, j, lane);
  } else if (wave >= nu) {
    for (int t = 0; t < NSTEP + D + 1; ++t) BARRIER();
  } else {
    f32x4 acc[2][4];
    for (int m = 0; m < 2; ++m)
      for (int n = 0; n < 4; ++n) acc[m][n] = f32x4{0.f, 0.f, 0.f, 0.f};
    f16x8 fa[2][2][2], fb[2][4][2];
    LDS_AS char* pa = smem + wave * 4096 + lane * 16;
    LDS_AS char* pb = smem + nu * 4096 + lane * 16;
    const unsigned sstep = nu * 4096 + 8192, send = sstep * (D + 1);
    unsigned slot = 0;
    auto rd = [&](auto sc) {
      constexpr int s = decltype(sc)::value;
      for (int u = 0; u < 2; ++u) {
        for (int m = 0; m < 2; ++m) fa[s][m][u] = *(LDS_AS f16x8*)(pa + slot + m * 2048 + u * 1024);
        for (int n = 0; n < 4; ++n) fb[s][n][u] = *(LDS_AS f16x8*)(pb + slot + u * 4096 + n * 1024);
      }
      slot += sstep;
      if (slot == send) slot = 0;
    };
    auto mm = [&](auto sc) {
      constexpr int s = decltype(sc)::value;
      for (int u = 0; u < 2; ++u)
        for (int m = 0; m < 2; ++m)
          for (int n = 0; n < 4; ++n)
            acc[m][n] = __builtin_amdgcn_mfma_f32_16x16x32_f16(fa[s][m][u], fb[s][n][u], acc[m][n], 0, 0, 0);
    };
    using I0 = std::integral_constant<int, 0>;
    using I1 = std::integral_constant<int, 1>;
    for (int t = 0; t <= D; ++t) BARRIER();
    rd(I0{});
    for (int s = 1; s < NSTEP - 1; s += 2) {
      BARRIER();
      rd(I1{});
      __builtin_amdgcn_sched_barrier(0);
      mm(I0{});
      __builtin_amdgcn_sched_barrier(0);
      BARRIER();
      rd(I0{});
      __builtin_amdgcn_sched_barrier(0);
      mm(I1{});
      __builtin_amdgcn_sched_barrier(0);
    }
    BARRIER();
    rd(I1{});
    const unsigned isbad = *bad;
    __builtin_amdgcn_sched_barrier(0);
    mm(I0{});
    __builtin_amdgcn_sched_barrier(0);
    BARRIER();
    mm(I1{});
    char* out = uniform(T.out + (size_t)wave * (T.staged ? 2 * 8192 : 32 * SHARD * 4));
    const unsigned vo = T.staged ? lane * 16 : (unsigned)(4 * (lane >> 4)) * (SHARD * 4) + (lane & 15) * 16;
    const unsigned sj = T.staged ? 1024 : SHARD * 4, sm = T.staged ? 4096 : 16 * SHARD * 4;
    for (int m = 0; m < 2; ++m) {
      for (int jj = 0; jj < 4; ++jj) {
        u32x4 o;
        for (int n = 0; n < 4; ++n)
          o[n] = isbad | __builtin_bit_cast(unsigned, gelu(acc[m][n][jj]));
        asm volatile("global_store_dwordx4 %1, %0, %2 sc0 sc1\n\ts_nop 1"
                     :: "v"(o), "v"(vo + m * sm + jj * sj), "s"(out) : "memory");
      }
    }
    if (T.oflag != nullptr) {
      wait_vm<0>();
      if (first_lane(lane)) flag_store(T.oflag + (T.staged ? flag_at(0, T.piece + 2 * wave) : wave * 256), call);
    }
    if (wave == 0) STAMP(dbg, 5);
  }
}

__device__ __forceinline__ Tile make_tile(const Params& P, int src, int u0, int nu, bool foreign, int xcd,
                                          unsigned call) {
  const int grp = P.rank >> 2, pos = P.rank & 3;
  const int c = xcd >> 1, ch = xcd & 1, cr = (4 - c) & 3;
  Tile T;
  int row0, R;
  bool direct = false;
  if (src < 4) {
    const int ub = src < 3 ? 256 * src : 512 + 64 * U2;
    T.au = src < 2 ? 4 : src == 2 ? U2 : UX;
    T.a = D_OWNX + ub * 4096;
    row0 = P.rank * SHARD + (ub >> 1);
    R = ub >> 6;
  } else if (src < 13) {
    const int i = (src - 4) / 3, b = src - 4 - 3 * i;
    T.au = b < 2 ? 4 : U2;
    T.a = D_INTRA + (3 * i + b) * MIB;
    row0 = (grp * 4 + ((pos + i + 1) & 3)) * SHARD + b * 128;
    R = b < 2 ? 8 + U2 + 12 * b + 4 * i : U2 * i;
    direct = b == 2;
  } else {
    const int k = src - 13;
    T.au = UX;
    T.a = D_CROSS + (3 * k + 1) * MIB;
    row0 = ((grp ^ 1) * 4 + ((pos + k) & 3)) * SHARD + ROWX;
    R = 32 + U2 + ((k - c - 1) & 3) * UX;
  }
  T.nu = nu;
  T.aflag = (src == 3 ? 3 : T.a >> 20) * 2048;
  T.a += u0 * 4096;
  row0 += u0 * 32;
  R += u0;
  T.staged = foreign && !direct;
  T.b = foreign ? D_CROSS + 3 * c * MIB : D_OWNW + c * MIB;
  T.bflag = (T.b >> 20) * 2048 + ch * 16;
  T.piece = 2 * R + ch;
  T.b += ch * 8192;
  if (!foreign) {
    T.out = (char*)P.out[P.rank & 7] + (size_t)row0 * (SHARD * 4) + xcd * 256;
    T.oflag = nullptr;
  } else {
    if (direct) {
      const int drank = ((grp ^ 1) * 4 + ((pos + c) & 3)) & 7;
      T.out = (char*)P.out[drank] + (size_t)row0 * (SHARD * 4) + cr * 512 + ch * 256;
      T.oflag = P.ws[drank] + OFF_OFLAG + (size_t)(cr * 64 + NWV + 2 * R + ch) * 128;
    } else {
      T.out = P.self + OFF_DATA + (size_t)(call & 1) * PARITY_B + D_STG + c * MIB + (2 * R + ch) * 8192;
      T.oflag = P.self + OFF_DONE + (call & 1) * 131072 + (D_STG / MIB + c) * 2048;
    }
  }
  return T;
}

__global__ void __launch_bounds__(NTHREADS, 1) ag_gemm_kernel(Params P) {
  __shared__ __attribute__((aligned(16))) char smem_raw[LDS_BYTES];
  LDS_AS char* smem = (LDS_AS char*)smem_raw;
  const int lane = threadIdx.x & 63;
  const int wave = __builtin_amdgcn_readfirstlane(threadIdx.x >> 6);
  const int bx = blockIdx.x, xcd = bx & 7, j = bx >> 3;
  const int grp = P.rank >> 2, pos = P.rank & 3;
  char* ws = P.self;
  unsigned* ctr = (unsigned*)(ws + OFF_CTR + (size_t)bx * 128);
  unsigned long long* dbg = (unsigned long long*)(ws + OFF_DBG + (size_t)bx * 512);
  if (wave == 9) STAMP(dbg, 0);

  const int sub = (xcd + j) & 7;
  if (j < JP && wave < 9) {
    const char* tick = ws + OFF_TICK + j * 256;
    if (wave >= PW) return;
    if (j < 3)
      send_stream<false>(P, (grp * 4 + ((pos + j + 1) & 3)) & 7, 0, D_INTRA / MIB + (2 - j) * 3, ctr, tick,
                         sub * PW + wave, lane, dbg + 16 * wave);
    else
      send_stream<true>(P, ((grp ^ 1) * 4 + ((pos + j - 3) & 3)) & 7, j - 3, D_CROSS / MIB + ((7 - j) & 3) * 3, ctr,
                        tick, sub * PW + wave, lane, dbg + 16 * wave);
    return;
  }
  const unsigned call = *(const CONST_AS unsigned*)(uintptr_t)ctr + 1u;
  asm volatile("" :: "s"(call));
  if (j < JP) {
    const int c = j - 3;
    BARRIER();
    if (j >= 3 && sub == 0) {
      const char* fp = ws + OFF_OFLAG + (size_t)(c * 64 + (lane < NWV + NDIR ? lane : 0)) * 128;
      char* tmo = ws + OFF_TMO;
      const unsigned long long t0 = __builtin_amdgcn_s_memrealtime();
      while (__builtin_amdgcn_s_memrealtime() - t0 < 4500) __builtin_amdgcn_s_sleep(32);
      for (;;) {
        const unsigned f = __hip_atomic_load((GLOBAL_AS unsigned*)fp, __ATOMIC_RELAXED, __HIP_MEMORY_SCOPE_SYSTEM);
        if (__all((int)(f - call) >= 0)) break;
        if (expired(tmo, call, t0, lane)) {
          if (first_lane(lane)) flag_store(P.out[P.rank & 7] + (size_t)P.rank * SHARD * SHARD, 0x7fc00000u);
          break;
        }
        __builtin_amdgcn_s_sleep(2);
      }
      STAMP(dbg, 62);
    }
  } else {
    const int r = j - JP;
    if (r >= 9 && wave < 8) {
      const int q = (r - 9) * 8 + xcd, t = q >> 1;
      unsigned own = 512 + (wave >> 1) * 128 + 2 * t + (wave & 1);
      if (!(q & 1)) {
        const int i = (t & 1) * 8 + wave, s = t >> 1;
        own = i < 8 ? (i >> 2) * 128 + 4 * s + (i & 3) : i < 8 + U2 ? 256 + U2 * s + i - 8
                                                                     : 256 + 32 * U2 + UX * s + i - 8 - U2;
      }
      const unsigned wait = (r < 9 + 2 * NH ? CONV_D0 : (r - 9 - 2 * NH) * CONV_DL) + (own >= 128 && own < 512 ? CONV_NU : 0);
      const unsigned long long t0 = __builtin_amdgcn_s_memrealtime();
      while (__builtin_amdgcn_s_memrealtime() - t0 < wait) __builtin_amdgcn_s_sleep(2);
      convert_chunk(P, own, call, (LDS_AS unsigned short*)(smem + wave * 1024), lane);
      if (wave == 0) STAMP(dbg, 40);
    }
    if (r >= 20) {
      const int w = wave * 5 + r - 20, link = w % 7, k = w / 7;
      if (k < UNP)
        unpack_stream(ws, link, call, (LDS_AS unsigned*)(smem + wave * 1024), k * 8 + ((xcd + link) & 7), lane,
                      wave == 0 ? dbg + 56 : wave == 5 ? dbg + 48 : dbg + 64 * 512);
    } else if (r >= 13 && wave == 9) {
      const unsigned long long t0 = __builtin_amdgcn_s_memrealtime();
      while (__builtin_amdgcn_s_memrealtime() - t0 < LATE) __builtin_amdgcn_s_sleep(16);
    }
    const int cx = 13 + (((xcd >> 1) + 1 + r % 3) & 3), i3 = 3 * (r % 3);
#define TD(src, u0, nu, foreign) ((src) | (u0) << 5 | (nu) << 7 | (foreign) << 10)
    int d0, d1 = 0, d2 = 0;
    if (r < 3) { d0 = TD(r, 0, r < 2 ? 4 : 3, 1); d1 = TD(5 + i3, 0, 2, 1); d2 = TD(6 + i3, 2, 1, 0); }
    else if (r < 6) { d0 = TD(4 + i3, 0, 4, 1); d1 = TD(cx, 3, 2, 1); }
    else if (r < 9) { d0 = TD(4 + i3, 0, 4, 0); d1 = TD(cx, 0, 3, 1); }
    else if (r < 12) { d0 = TD(r - 9, 0, r < 11 ? 4 : 3, 0); d1 = TD(5 + i3, 2, 2, 1); d2 = TD(6 + i3, 0, 2, 0); }
    else if (r == 12) { d0 = TD(3, 0, 5, 0); d1 = TD(6, 2, 1, 1); }
    else if (r < 16) d0 = TD(5 + i3, 0, 4, 0);
    else if (r < 20) d0 = TD(r - 3, 0, 5, 0);
    else if (r < 23) d0 = TD(6 + i3, 0, 2, 1);
    else d0 = TD(6 + 3 * (r - 22), 2, 1, 1);
#pragma unroll 1
    for (int n = 0; d0 != 0; ++n, d0 = d1, d1 = d2, d2 = 0) {
      const Tile T = make_tile(P, d0 & 31, (d0 >> 5) & 3, (d0 >> 7) & 7, d0 >> 10, xcd, call);
      gemm_tile(smem, ws, T, call, wave, lane, dbg + 8 + 8 * n);
    }
  }
  if (wave == 9) {
    if (first_lane(lane)) __hip_atomic_store((GLOBAL_AS unsigned*)ctr, call, __ATOMIC_RELAXED, __HIP_MEMORY_SCOPE_AGENT);
    STAMP(dbg, 63);
  }
}

extern "C" void kernel_launch(int rank, int nranks, ncclComm_t comm,
                              hipStream_t stream,
                              void* const* d_in, const int* in_sizes, int n_in,
                              void* d_out, int out_size,
                              const int* in_dtypes, int out_dtype,
                              void* const* d_out_all,
                              void* const* d_ws_all, size_t ws_bytes) {
  (void)comm; (void)in_dtypes; (void)out_dtype; (void)d_out;
  if (nranks != NRANK || (unsigned)rank >= (unsigned)NRANK || n_in < 2 || ws_bytes < WS_NEED ||
      in_sizes[0] != SHARD * KDIM || in_sizes[1] != KDIM * SHARD || out_size != NRANK * SHARD * SHARD) {
    fprintf(stderr, "unexpected problem shape\n");
    return;
  }
  Params P;
  P.x = (const float*)d_in[0];
  P.w = (const float*)d_in[1];
  for (int p = 0; p < NRANK; ++p) {
    P.ws[p] = (char*)d_ws_all[p];
    P.out[p] = (float*)d_out_all[p];
  }
  P.self = (char*)d_ws_all[rank];
  P.rank = rank;
  P.pad = 0;
  ag_gemm_kernel<<<NWG, NTHREADS, 0, stream>>>(P);
  HIP_CHECK(hipGetLastError());
}
```

```cpp
#include <hip/hip_runtime.h>
#include <rccl/rccl.h>
#include <cstdio>
#include <cstdint>
#include <utility>

#define HIP_CHECK(expr) do { hipError_t _e = (expr); if (_e != hipSuccess) { \
    fprintf(stderr, "HIP error %s at %s:%d\n", hipGetErrorString(_e), __FILE__, __LINE__); \
    return; } } while (0)

typedef unsigned u32x4 __attribute__((ext_vector_type(4)));
typedef unsigned u32x2 __attribute__((ext_vector_type(2)));
typedef float f32x4 __attribute__((ext_vector_type(4)));
typedef _Float16 f16x8 __attribute__((ext_vector_type(8)));
#define LDS_AS __attribute__((address_space(3)))
#define CONST_AS __attribute__((address_space(4)))
#define GLOBAL_AS __attribute__((address_space(1)))

#define U2 3
#define PW 2
#define NH 4
#define S1P 2
#define S1R 0
#define NEAR 10
#define EVERY 4
#define AGAIN 1
#define RT1 0
#define RTX 0
#define POLL_SLEEP 4
#define TOP_LANES 8
#define UNP_SLEEP 4
#define FORCE_RAW 0
#define CONV_D0 100
#define CONV_DL 0
#define SND_SLEEP 4
#define AHEAD 64
#define LATE 1000
#define UNP 7
#define CONV_NU 300

constexpr int NRANK = 8;
constexpr int KDIM = 4096;
constexpr int SHARD = 512;
constexpr int UX = 8 - U2;
constexpr int ROWX = 256 + 32 * U2;
constexpr int NWV = 8 * PW;
constexpr int NHEAD = NH * NWV;
constexpr int JP = 7;
constexpr int NWG = 256;
constexpr int NTHREADS = 640;
constexpr unsigned MIB = 1u << 20;
constexpr int NSTEP = 64;
constexpr int NSTG = 2 * (32 + U2 + 3 * UX);
constexpr int NDIR = 2 * 3 * U2;

constexpr size_t OFF_CTR = 0;
constexpr size_t OFF_TMO = 32768;
constexpr size_t OFF_TICK = 36864;
constexpr size_t OFF_OFLAG = 131072;
constexpr size_t OFF_DBG = 262144;
constexpr size_t OFF_DONE = 786432;
constexpr size_t OFF_DATA = MIB;
constexpr size_t PARITY_B = 62 * MIB;
constexpr size_t D_OWNX = 0, D_OWNW = 4 * MIB, D_INTRA = 8 * MIB, D_CROSS = 17 * MIB, D_STG = 29 * MIB;
constexpr size_t PK_OFF = 25 * MIB;
constexpr size_t D_IMG = 54 * MIB;
constexpr unsigned PK_FLAG = 7680;
constexpr int NUW = 8 * UNP;
constexpr size_t WS_NEED = OFF_DATA + 2 * PARITY_B;
static_assert(D_CROSS == D_INTRA + 9 * MIB && D_INTRA + PK_OFF >= D_STG + 4 * MIB && D_STG + PK_OFF <= D_IMG &&
              D_IMG + 8 * MIB <= PARITY_B && NSTG <= 128 && NWV + NDIR <= 64 && U2 == 3 && UNP <= 7, "layout");

constexpr unsigned long long DEADLINE_TICKS = 40000000ull;

constexpr int LDS_RING = 147456;
constexpr int LDS_BYTES = LDS_RING + 64 + 1024;

struct Params {
  const float* x;
  const float* w;
  char* self;
  char* ws[NRANK];
  float* out[NRANK];
  int rank;
  int pad;
};

template <int... I, class F>
__device__ __forceinline__ void static_for_impl(std::integer_sequence<int, I...>, F&& f) {
  (f(std::integral_constant<int, I>{}), ...);
}
template <int N, class F>
__device__ __forceinline__ void static_for(F&& f) {
  static_for_impl(std::make_integer_sequence<int, N>{}, f);
}

#define BARRIER() do { asm volatile("" ::: "memory"); __builtin_amdgcn_s_waitcnt(0xC07F); \
    asm volatile("s_barrier" ::: "memory"); } while (0)

#define STAMP(dbg, id) do { } while (0)

__device__ __forceinline__ unsigned pk16(float a, float b) {
  unsigned r;
  asm("v_cvt_pk_f16_f32 %0, %1, %2" : "=v"(r) : "v"(a), "v"(b));
  return r;
}

__device__ __forceinline__ bool first_lane(int lane) {
  asm volatile("" : "+v"(lane));
  return lane == 0;
}
__device__ __forceinline__ char* uniform(char* p) {
  const unsigned long long a = (unsigned long long)p;
  unsigned lo = __builtin_amdgcn_readfirstlane((unsigned)a), hi = __builtin_amdgcn_readfirstlane((unsigned)(a >> 32));
  asm volatile("s_nop 4" : "+s"(lo), "+s"(hi));
  return (char*)((unsigned long long)hi << 32 | lo);
}
__device__ __forceinline__ void flag_store(void* p, unsigned v) {
  __hip_atomic_store((GLOBAL_AS unsigned*)p, v, __ATOMIC_RELAXED, __HIP_MEMORY_SCOPE_SYSTEM);
}
__device__ __forceinline__ bool expired(char* tmo, unsigned call, unsigned long long t0, int lane) {
  const unsigned long long dt = __builtin_amdgcn_s_memrealtime() - t0;
  if (dt < 10000) return false;
  const unsigned tv = __hip_atomic_load((GLOBAL_AS unsigned*)tmo, __ATOMIC_RELAXED, __HIP_MEMORY_SCOPE_AGENT);
  if (__builtin_amdgcn_readfirstlane(tv) == call || dt > DEADLINE_TICKS) {
    if (first_lane(lane)) __hip_atomic_store((GLOBAL_AS unsigned*)tmo, call, __ATOMIC_RELAXED, __HIP_MEMORY_SCOPE_AGENT);
    return true;
  }
  return false;
}

__device__ __forceinline__ void load_unit(f32x4 (&b)[8], const unsigned (&v)[8], const char* s) {
  asm volatile("global_load_dwordx4 %0, %8, %16\n\t"
               "global_load_dwordx4 %1, %9, %16\n\t"
               "global_load_dwordx4 %2, %10, %16\n\t"
               "global_load_dwordx4 %3, %11, %16\n\t"
               "global_load_dwordx4 %4, %12, %16\n\t"
               "global_load_dwordx4 %5, %13, %16\n\t"
               "global_load_dwordx4 %6, %14, %16\n\t"
               "global_load_dwordx4 %7, %15, %16"
               : "=&v"(b[0]), "=&v"(b[1]), "=&v"(b[2]), "=&v"(b[3]), "=&v"(b[4]), "=&v"(b[5]), "=&v"(b[6]), "=&v"(b[7])
               : "v"(v[0]), "v"(v[1]), "v"(v[2]), "v"(v[3]), "v"(v[4]), "v"(v[5]), "v"(v[6]), "v"(v[7]), "s"(s)
               : "memory");
}
template <int CNT>
__device__ __forceinline__ void wait_unit(f32x4 (&b)[8]) {
  asm volatile("s_waitcnt vmcnt(%8)"
               : "+v"(b[0]), "+v"(b[1]), "+v"(b[2]), "+v"(b[3]), "+v"(b[4]), "+v"(b[5]), "+v"(b[6]), "+v"(b[7])
               : "i"(CNT) : "memory");
}
template <int CNT>
__device__ __forceinline__ void wait_vm() {
  asm volatile("s_waitcnt vmcnt(%0)" :: "i"(CNT) : "memory");
}
__device__ __forceinline__ void cvt_unit(unsigned* d, const f32x4 (&b)[8], bool isw) {
  if (isw) {
    for (int n = 0; n < 4; ++n)
      for (int e = 0; e < 4; ++e) d[4 * n + e] = pk16(b[2 * e][n], b[2 * e + 1][n]);
  } else {
    for (int u = 0; u < 8; ++u) {
      d[2 * u] = pk16(b[u][0], b[u][1]);
      d[2 * u + 1] = pk16(b[u][2], b[u][3]);
    }
  }
}
__device__ __forceinline__ void store_4k(const u32x4 (&d)[4], unsigned v, char* p, unsigned long long tail = ~0ull) {
  asm volatile("global_store_dwordx4 %4, %0, %5 sc0 sc1\n\t"
               "global_store_dwordx4 %4, %1, %5 offset:1024 sc0 sc1\n\t"
               "global_store_dwordx4 %4, %2, %5 offset:2048 sc0 sc1\n\t"
               "s_mov_b64 exec, %6\n\t"
               "global_store_dwordx4 %4, %3, %5 offset:3072 sc0 sc1\n\t"
               "s_mov_b64 exec, -1\n\ts_nop 0"
               :: "v"(d[0]), "v"(d[1]), "v"(d[2]), "v"(d[3]), "v"(v), "s"(p), "s"(tail) : "memory");
}
__device__ __forceinline__ void gather(u32x4 (&q)[4], const unsigned* d) {
  for (int k = 0; k < 4; ++k) q[k] = u32x4{d[4 * k], d[4 * k + 1], d[4 * k + 2], d[4 * k + 3]};
}
__device__ __forceinline__ void store_4k(const unsigned* d, unsigned v, char* p, unsigned long long tail = ~0ull) {
  u32x4 q[4];
  gather(q, d);
  store_4k(q, v, p, tail);
}
__device__ __forceinline__ void unit_offsets(unsigned (&v)[8], bool isw, int lane) {
  const int q = lane >> 4, c = lane & 15;
  for (int j = 0; j < 8; ++j)
    v[j] = isw ? (unsigned)((j >> 2) * 16 + 4 * q + (j & 3)) * (SHARD * 4) + c * 16
               : (unsigned)((j >> 2) * 16 + c) * (KDIM * 4) + (j & 3) * 64 + q * 16;
}
__device__ __forceinline__ unsigned w_unit(unsigned n) {
  return ((n >> 2) * 2 + (n & 1)) * (32 * SHARD * 4) + ((n >> 1) & 1) * 256;
}
__device__ __forceinline__ unsigned x_unit(unsigned n) {
  unsigned step, un;
  if (n < 512) { step = (n & 255) >> 2; un = (n >> 8) * 4 + (n & 3); }
  else if (n < 512 + 64 * U2) { step = (n - 512) / U2; un = 8 + (n - 512) % U2; }
  else { step = (n - 512 - 64 * U2) / UX; un = 8 + U2 + (n - 512 - 64 * U2) % UX; }
  return un * (32 * KDIM * 4) + step * 256;
}

__device__ __forceinline__ unsigned w_flag(unsigned c) { return (c & 1) * 64 + (c >> 1); }
__device__ __forceinline__ unsigned flag_at(unsigned block, unsigned i) { return block * 2048 + (i & 15) * 128 + (i >> 4) * 4; }
__device__ __forceinline__ unsigned stream_flag(bool cross, unsigned c) {
  return !cross ? flag_at(c >> 7, c & 127) : c < 128 ? flag_at(0, w_flag(c)) : flag_at(1, c - 128);
}

__device__ __forceinline__ int first_unset(u32x4 f, unsigned call, int at) {
  int now = 4096;
  for (int k = 3; k >= 0; --k) now = (int)(f[k] - call) >= 0 ? now : at + 16 * k;
  return now;
}
__device__ __forceinline__ void look(int& first, bool on, const char* fp, unsigned call, int at) {
  u32x4 f;
  on = on && first != 4096;
  asm volatile("s_mov_b64 exec, %2\n\t"
               "global_load_dwordx4 %0, %1, off sc0 sc1\n\t"
               "s_mov_b64 exec, -1\n\ts_waitcnt vmcnt(0)"
               : "=&v"(f) : "v"(fp), "s"(__ballot(on)) : "memory");
  first = on ? first_unset(f, call, at) : first;
}

__device__ __forceinline__ unsigned pk_add(unsigned a, unsigned b) {
  typedef unsigned short u16x2 __attribute__((ext_vector_type(2)));
  return __builtin_bit_cast(unsigned, (u16x2)(__builtin_bit_cast(u16x2, a) + __builtin_bit_cast(u16x2, b)));
}
__device__ __forceinline__ unsigned bfi(unsigned mask, unsigned a, unsigned b) { return (a & mask) | (b & ~mask); }
constexpr unsigned HOLE = 0x60006000u;

__device__ __forceinline__ void encode_chunk(unsigned (&m)[32], unsigned (&x)[32], float mx,
                                                 LDS_AS unsigned short* list, int lane) {
  const unsigned le = pk16(mx, mx) >> 26;
  unsigned top = 0;
  for (unsigned bit = 16; bit; bit >>= 1) {
    unsigned above = __popcll(__ballot(le >= top + bit));
    asm("" : "+s"(above));
    if (above > TOP_LANES) top += bit;
  }
  top = top < 7 ? 7 : top;
  asm("" : "+s"(top));
  const unsigned base = top - 7;
  const unsigned down = ((64 - base) & 63) * 0x04000400u;
  unsigned P[4];
  for (int g = 0; g < 4; ++g) {
    for (int i = 0; i < 8; ++i) x[8 * g + i] = pk_add(x[8 * g + i], down);
    P[g] = x[8 * g + 7] << 1;
    for (int i = 0; i < 7; ++i) P[g] = bfi(0x00030003u << (2 * i), x[8 * g + i] >> (13 - 2 * i), P[g]);
  }
  unsigned n = 0;
#pragma unroll 1
  do {
    for (int g = 0; g < 4; ++g) {
      const unsigned p = P[g];
      const unsigned long long mask = __ballot(p != 0);
      const unsigned sh = __builtin_ctz(p | 0x80000000u) & 30;
      const unsigned at = __builtin_amdgcn_mbcnt_hi((unsigned)(mask >> 32), __builtin_amdgcn_mbcnt_lo((unsigned)mask, 0));
      *(volatile LDS_AS unsigned short*)(list + n + (p != 0 ? at : 64)) = (lane * 4 + g) << 6 | sh << 1 | ((p >> sh) & 3);
      P[g] = p & ~(3u << sh);
      n += __popcll(mask);
    }
  } while (__ballot((P[0] | P[1] | P[2] | P[3]) != 0) != 0 && n <= 128);
  const u32x4 esc = *(volatile LDS_AS u32x4*)(list + 8 * (lane & 15));
  for (int g = 0; g < 4; ++g) {
    const unsigned y = bfi(0x20002000u, x[8 * g + 7] >> 2, x[8 * g + 7]);
    for (int i = 0; i < 7; ++i) m[7 * g + i] = bfi(HOLE, y << (13 - 2 * i), x[8 * g + i]);
    m[28 + g] = esc[g];
  }
  if (lane >= 16) m[28] = FORCE_RAW || n > 128 ? 32 : base | n << 8;
}

__device__ __forceinline__ void decode_half(unsigned (&d)[16], const unsigned* m, unsigned p0, unsigned p1,
                                            unsigned base) {
  const unsigned up = base * 0x04000400u;
  for (int g = 0; g < 2; ++g) {
    const unsigned p = g ? p1 : p0;
    unsigned y = m[7 * g] >> 13;
    for (int i = 0; i < 7; ++i) {
      if (i) y = bfi(0x00030003u << (2 * i), m[7 * g + i] >> (13 - 2 * i), y);
      d[8 * g + i] = pk_add(bfi(HOLE, p << (13 - 2 * i), m[7 * g + i]), up);
    }
    d[8 * g + 7] = pk_add(bfi(HOLE, p >> 1, bfi(0x80008000u, y << 2, y)), up);
  }
}

__device__ __forceinline__ int staged_row(int R, int c, int rank) {
  const int grp = rank >> 2, pos = rank & 3;
  if (R < 8 + U2) return rank * SHARD + R * 32;
  R -= 8 + U2;
  if (R < 24) {
    const int b = R >= 12 ? 1 : 0, m = R - 12 * b;
    return (grp * 4 + ((pos + 1 + (m >> 2)) & 3)) * SHARD + b * 128 + (m & 3) * 32;
  }
  R -= 24;
  const int k = R / UX;
  return ((grp ^ 1) * 4 + ((pos + c + 1 + k) & 3)) * SHARD + ROWX + (R - k * UX) * 32;
}

__device__ __forceinline__ void store_rows(const u32x4 (&d)[4], unsigned v, char* s) {
  asm volatile("global_store_dwordx4 %4, %0, %5 offset:-4096 sc0 sc1\n\t"
               "global_store_dwordx4 %4, %1, %5 offset:-2048 sc0 sc1\n\t"
               "global_store_dwordx4 %4, %2, %5 sc0 sc1\n\t"
               "global_store_dwordx4 %4, %3, %5 offset:2048 sc0 sc1\n\ts_nop 1"
               :: "v"(d[0]), "v"(d[1]), "v"(d[2]), "v"(d[3]), "v"(v), "s"(s) : "memory");
}
__device__ __forceinline__ void load_4k(u32x4 (&d)[4], unsigned v, const char* s) {
  asm volatile("global_load_dwordx4 %0, %4, %5 sc0 sc1\n\t"
               "global_load_dwordx4 %1, %4, %5 offset:1024 sc0 sc1\n\t"
               "global_load_dwordx4 %2, %4, %5 offset:2048 sc0 sc1\n\t"
               "global_load_dwordx4 %3, %4, %5 offset:3072 sc0 sc1"
               : "+v"(d[0]), "+v"(d[1]), "+v"(d[2]), "+v"(d[3]) : "v"(v), "s"(s) : "memory");
}
template <int CNT>
__device__ __forceinline__ void wait_4k(u32x4 (&d)[4]) {
  asm volatile("s_waitcnt vmcnt(%4)" : "+v"(d[0]), "+v"(d[1]), "+v"(d[2]), "+v"(d[3]) : "i"(CNT) : "memory");
}
__device__ __forceinline__ bool packed(bool cross, unsigned c) { return c >= NHEAD && c < (cross ? 32 * (4 + UX) - RTX : 256 - (RT1)); }

template <bool CROSS>
__device__ __forceinline__ void send_stream(const Params& P, int peer, int c, unsigned block, const unsigned* ctr,
                                            const char* tick, int v, int lane, unsigned long long* dbg) {
  constexpr unsigned NCH = 32 * (CROSS ? 4 + UX : 8 + U2);
  constexpr unsigned TOT = NCH + (CROSS ? NSTG : 0);
  const unsigned vs = lane * 16;
  u32x4 A[4], B[4];
  f32x4 b0[8], b1[8];
  unsigned voff[8], t[16];
  unit_offsets(voff, CROSS, lane);
  const char* src = CROSS ? (const char*)P.w + c * 512 : (const char*)P.x;
  __builtin_amdgcn_s_waitcnt(0x0F70);
  load_unit(b0, voff, src + (CROSS ? w_unit(2 * v) : x_unit(2 * v)));
  load_unit(b1, voff, src + (CROSS ? w_unit(2 * v + 1) : x_unit(2 * v + 1)));
  const unsigned call = *(const CONST_AS unsigned*)(uintptr_t)ctr + 1u;
  asm volatile("" :: "s"(call));
  BARRIER();
  const unsigned par = call & 1;
  const int cr = (4 - c) & 3;
  char* dst = P.ws[peer] + OFF_DATA + (size_t)par * PARITY_B + (size_t)block * MIB;
  char* done = P.ws[peer] + OFF_DONE + par * 131072 + block * 2048;
  unsigned cur = v, nxt = NWV + v;
  wait_unit<8>(b0);
  STAMP(dbg, 15);
  static_for<NH - 1>([&](auto hc) {
    cvt_unit(t, b0, CROSS);
    store_4k(t, vs, dst + cur * 8192);
    load_unit(b0, voff, src + (CROSS ? w_unit(2 * nxt) : x_unit(2 * nxt)));
    wait_unit<8>(b1);
    cvt_unit(t, b1, CROSS);
    store_4k(t, vs + 4096, dst + cur * 8192);
    load_unit(b1, voff, src + (CROSS ? w_unit(2 * nxt + 1) : x_unit(2 * nxt + 1)));
    wait_unit<8>(b0);
    STAMP(dbg, 1 + (cur >> 5));
    cur = nxt;
    nxt += NWV;
    if (decltype(hc)::value < NH - 2 && first_lane(lane)) flag_store(done + stream_flag(CROSS, cur - NWV), call);
  });
  cvt_unit(t, b0, CROSS);
  gather(A, t);
  wait_unit<0>(b1);
  if (first_lane(lane)) flag_store(done + stream_flag(CROSS, cur - NWV), call);
  cvt_unit(t, b1, CROSS);
  gather(B, t);

  const char* mine = P.self + OFF_DONE + par * 131072 + (lane & 15) * 128 + (lane >> 4) * 16;
  char* out = (char*)P.out[peer] + cr * 512;
  char* data = P.self + OFF_DATA + (size_t)par * PARITY_B;
  char* tmo = P.self + OFF_TMO;
  const unsigned vo = (unsigned)(4 * (lane >> 4) + 2) * (SHARD * 4) + (lane & 15) * 16;
  const unsigned base = (call - 1) * (TOT - NHEAD) - NHEAD - NWV;
  unsigned info = 32, seen = ~0u;
  int first = 0;
#pragma unroll 1
  for (;;) {
    const bool last = nxt >= TOT, piece = CROSS && cur >= NCH;
    const unsigned e = cur - NCH;
    char* to = piece ? out + (size_t)staged_row(e >> 1, c, P.rank) * (SHARD * 4) + (e & 1) * 256
                     : dst + cur * 8192 + (info == 32 ? 0 : PK_OFF);
    if (piece) store_rows(A, vo, to); else store_4k(A, vs, to);
    unsigned tk, inf;
    asm volatile("s_mov_b64 exec, 1\n\t"
                 "global_atomic_add %0, %1, %2, %3 sc0\n\t"
                 "s_mov_b64 exec, -1"
                 : "=&v"(tk) : "v"(0u), "v"(last ? 0u : 1u), "s"(tick) : "memory");
    unsigned from = nxt, blk = nxt >> 7, at = nxt & 127;
    if (CROSS) {
      if (nxt < 128) { from = 512 + 128 * c + nxt; blk = 4 + c; at = w_flag(nxt); }
      else if (nxt < NCH) { from = nxt + 128 + 32 * U2; blk = 3; at = nxt - 128; }
      else { blk = D_STG / MIB + c; at = nxt - NCH; from = blk * 128 + at; }
    }
    const int who = (at & 15) | (at >> 6) << 4;
    bool ok = false;
    if (!last) {
      if (blk != seen) { seen = blk; first = 0; }
      if (__builtin_amdgcn_readlane(first, who) <= (int)at)
        look(first, true, mine + blk * 2048, call, (lane & 15) + (lane >> 4) * 64);
      ok = __builtin_amdgcn_readlane(first, who) > (int)at;
    }
    bool image = packed(CROSS, nxt);
    const char* next = uniform(data + (ok ? (image ? (unsigned)D_IMG : 0u) + from * 8192 : 0u));
    load_4k(A, vs, next);
    asm volatile("global_load_dword %0, %1, %2 sc0 sc1" : "=&v"(inf) : "v"(7168u + 256u), "s"(next) : "memory");
    if (S1P != S1R && info == 32) wait_vm<6 + S1R>();
    wait_4k<6 + S1P>(B);
    u32x4 f;
    if (AGAIN)
      asm volatile("s_mov_b64 exec, %2\n\t"
                   "global_load_dwordx4 %0, %1, off sc0 sc1\n\t"
                   "s_mov_b64 exec, -1"
                   : "=&v"(f) : "v"(mine + blk * 2048), "s"(ok || last ? 0ull : 1ull << who) : "memory");
    if (piece) store_rows(B, vo + 16 * (SHARD * 4), to);
    else store_4k(B, vs + 4096, to, info == 32 ? ~0ull : info >= (97 << 8) ? 0xffff : info >= (65 << 8) ? 0xfff
                                                       : info >= (33 << 8) ? 0xff : 0xf);
    if (AGAIN && !ok && !last) {
      asm volatile("s_waitcnt vmcnt(4)" : "+v"(f), "+v"(inf) :: "memory");
      first = lane == who ? first_unset(f, call, (lane & 15) + (lane >> 4) * 64) : first;
      ok = __builtin_amdgcn_readlane(first, who) > (int)at;
      if (ok) {
        next = uniform(data + (image ? (unsigned)D_IMG : 0u) + from * 8192);
        load_4k(A, vs, next);
        asm volatile("global_load_dword %0, %1, %2 sc0 sc1" : "+v"(inf) : "v"(7168u + 256u), "s"(next) : "memory");
      }
    }
    load_4k(B, vs + 4096, next);
    wait_4k<4>(A);
    asm volatile("" : "+v"(tk), "+v"(inf));
    if (AGAIN) asm volatile("" : "+v"(f));
    if (!piece && first_lane(lane)) {
      if (info == 32) flag_store(done + stream_flag(CROSS, cur), call);
      if (packed(CROSS, cur))
        __hip_atomic_store((GLOBAL_AS unsigned long long*)(dst + cur * 8192 + PK_OFF + PK_FLAG),
                           (unsigned long long)call << 32 | info, __ATOMIC_RELAXED, __HIP_MEMORY_SCOPE_SYSTEM);
    }
    STAMP(dbg, 1 + (cur >> 5));
    if (last) break;
    cur = nxt;
    nxt = __builtin_amdgcn_readfirstlane(tk) - base;
    info = image ? __builtin_amdgcn_readfirstlane(inf) : 32;
    if (!ok || (image && info == 32)) {
      const unsigned long long t0 = __builtin_amdgcn_s_memrealtime();
      bool dead = false;
      while (__builtin_amdgcn_readlane(first, who) <= (int)at) {
        look(first, lane == who, mine + blk * 2048, call, (lane & 15) + (lane >> 4) * 64);
        if (__builtin_amdgcn_readlane(first, who) > (int)at) break;
        if (expired(tmo, call, t0, lane)) { dead = true; break; }
        __builtin_amdgcn_s_sleep(SND_SLEEP);
      }
      wait_4k<0>(B);
#pragma unroll 1
      for (image = image && !ok;; image = false) {
        const char* again = uniform(data + (image ? (unsigned)D_IMG : 0u) + from * 8192);
        load_4k(A, vs, again);
        load_4k(B, vs + 4096, again);
        wait_4k<0>(A);
        wait_4k<0>(B);
        info = image ? __builtin_amdgcn_readlane(B[3][0], 16) : 32;
        if (!image || info != 32) break;
      }
      if (dead) {
        for (int k = 0; k < 4; ++k) A[k] = B[k] = u32x4{0x7fc07fc0u, 0x7fc07fc0u, 0x7fc07fc0u, 0x7fc07fc0u};
        info = 32;
      }
    }
  }
  wait_4k<0>(B);
  if (CROSS && first_lane(lane)) flag_store(P.ws[peer] + OFF_OFLAG + (size_t)(cr * 64 + v) * 128, call);
  STAMP(dbg, 14);
}

__device__ __forceinline__ void unpack_stream(char* ws, int link, unsigned call, LDS_AS unsigned* plane, int u,
                                              int lane, unsigned long long* dbg) {
  const unsigned block = D_INTRA / MIB + link * 3;
  const unsigned end = link < 3 ? 256 - (RT1) : 32 * (4 + UX) - RTX;
  char* whole = ws + OFF_DATA + (size_t)(call & 1) * PARITY_B + (size_t)block * MIB;
  char* done = ws + OFF_DONE + (call & 1) * 131072 + block * 2048;
  char* tmo = ws + OFF_TMO;
  const unsigned vs = lane * 16;
  unsigned owed = ~0u;
#pragma unroll 1
  for (unsigned cur = NHEAD + u; cur < end; cur += NUW) {
    const char* slot = whole + cur * 8192 + PK_OFF;
    *(volatile LDS_AS u32x4*)(plane + 4 * lane) = u32x4{0, 0, 0, 0};
    unsigned info = 32;
    const unsigned long long t0 = __builtin_amdgcn_s_memrealtime();
    for (;;) {
      u32x2 f;
      asm volatile("global_load_dwordx2 %0, %1, %2 sc0 sc1\n\ts_waitcnt vmcnt(0)"
                   : "=&v"(f) : "v"(PK_FLAG), "s"(slot) : "memory");
      if (owed != ~0u) {
        if (first_lane(lane)) flag_store(done + owed, call);
        owed = ~0u;
      }
      if ((int)(__builtin_amdgcn_readfirstlane(f[1]) - call) >= 0) { info = __builtin_amdgcn_readfirstlane(f[0]); break; }
      if (expired(tmo, call, t0, lane)) break;
      __builtin_amdgcn_s_sleep(UNP_SLEEP);
    }
    if (info == 32) continue;
    u32x4 q[7];
    unsigned esc;
    asm volatile("global_load_dword %7, %9, %10 offset:3072 sc0 sc1\n\t"
                 "global_load_dwordx4 %0, %8, %10 offset:-4096 sc0 sc1\n\t"
                 "global_load_dwordx4 %1, %8, %10 offset:-3072 sc0 sc1\n\t"
                 "global_load_dwordx4 %2, %8, %10 offset:-2048 sc0 sc1\n\t"
                 "global_load_dwordx4 %3, %8, %10 offset:-1024 sc0 sc1\n\t"
                 "global_load_dwordx4 %4, %8, %10 sc0 sc1\n\t"
                 "global_load_dwordx4 %5, %8, %10 offset:1024 sc0 sc1\n\t"
                 "global_load_dwordx4 %6, %8, %10 offset:2048 sc0 sc1\n\t"
                 "s_waitcnt vmcnt(0)"
                 : "=&v"(q[0]), "=&v"(q[1]), "=&v"(q[2]), "=&v"(q[3]), "=&v"(q[4]), "=&v"(q[5]), "=&v"(q[6]),
                   "=&v"(esc)
                 : "v"(vs), "v"(vs >> 2), "s"(slot + 4096) : "memory");
    const unsigned n = info >> 8;
    for (int h = 0; h < 2; ++h) {
      const unsigned e = esc >> (16 * h);
      const unsigned val = 2 * lane + h < n ? (e & 3) << ((e >> 1) & 30) : 0;
      __hip_atomic_fetch_or(plane + ((e >> 6) & 255), val, __ATOMIC_RELAXED, __HIP_MEMORY_SCOPE_WORKGROUP);
    }
    asm volatile("" ::: "memory");
    const u32x4 P = *(volatile LDS_AS u32x4*)(plane + 4 * lane);
    unsigned m[28];
    for (int k = 0; k < 28; ++k) m[k] = q[k >> 2][k & 3];
    for (int h = 0; h < 2; ++h) {
      unsigned d[16];
      decode_half(d, m + 14 * h, P[2 * h], P[2 * h + 1], info & 31);
      store_4k(d, vs, whole + cur * 8192 + 4096 * h);
    }
    owed = stream_flag(link >= 3, cur);
  }
  wait_vm<0>();
  if (owed != ~0u && first_lane(lane)) flag_store(done + owed, call);
  STAMP(dbg, 3);
}

__device__ __forceinline__ void convert_chunk(const Params& P, unsigned own, unsigned call, LDS_AS unsigned short* list,
                                              int lane) {
  const bool isw = own >= 512;
  const char* src = isw ? (const char*)P.w + ((own - 512) >> 7) * 512 : (const char*)P.x;
  char* data = P.self + OFF_DATA + (size_t)(call & 1) * PARITY_B + (size_t)own * 8192;
  f32x4 buf[2][8];
  unsigned voff[8], x[32];
  unit_offsets(voff, isw, lane);
  load_unit(buf[0], voff, src + (isw ? w_unit(2 * own & 255) : x_unit(2 * own)));
  load_unit(buf[1], voff, src + (isw ? w_unit((2 * own + 1) & 255) : x_unit(2 * own + 1)));
  wait_unit<8>(buf[0]);
  wait_unit<0>(buf[1]);
  float mx = 0.f;
  for (int h = 0; h < 2; ++h) {
    cvt_unit(x + 16 * h, buf[h], isw);
    store_4k(x + 16 * h, lane * 16, data + 4096 * h);
    for (int j = 0; j < 8; j += 2)
      asm("v_max3_f32 %0, |%1|, |%2|, %0" : "+v"(mx) : "v"(buf[h][j][0]), "v"(buf[h][j + 1][3]));
  }
  if (isw ? (own & 127) >= NHEAD : own < 256 + 32 * U2 ? packed(false, own) : packed(true, own - 128 - 32 * U2)) {
    unsigned m[32];
    encode_chunk(m, x, mx, list, lane);
    store_4k(m, lane * 16, data + D_IMG);
    store_4k(m + 16, lane * 16, data + D_IMG + 4096, 0xfffffull);
  }
  wait_vm<0>();
  const unsigned at = isw ? flag_at(own >> 7, w_flag(own & 127))
                          : own < 256 + 32 * U2 ? flag_at(own >> 7, own & 127) : flag_at(3, own - 256 - 32 * U2);
  if (first_lane(lane)) flag_store(P.self + OFF_DONE + (call & 1) * 131072 + at, call);
}

__device__ __forceinline__ float gelu(float y) {
  const float k1 = -2.0f * 0.7978845608028654f * 1.4426950408889634f;
  const float k3 = k1 * 0.044715f;
  float e = __builtin_amdgcn_exp2f(y * (k1 + k3 * y * y));
  return y * __builtin_amdgcn_rcpf(1.0f + e);
}

#define SMALL 2
#define ASC " sc0 sc1"
#define BSC " sc0 sc1"
#define NS1 12
#define NS2 9
__host__ __device__ constexpr int ring_slots(int nu) { return nu == 5 ? 5 : nu == 4 ? 6 : nu == 3 ? 7 : nu == 2 ? NS2 : NS1; }

struct Tile {
  unsigned a, b;
  unsigned aflag, bflag;
  int au, nu;
  char* out;
  char* oflag;
  int piece;
  bool staged;
};

template <int NU>
__device__ __forceinline__ void tile_loader(unsigned lbase, const char* a, unsigned astep, const char* b, int j,
                                            int lane) {
  constexpr int SLOT = NU * 4096 + 8192;
  constexpr int NS = ring_slots(NU);
  constexpr int D = NS - 1;
  constexpr int NLD = NU + 2;
  static_assert(NS * SLOT <= LDS_RING && (D - 1) * NLD <= 63, "ring");
  const unsigned voff = lane * 16;
  a += j * (NU * 1024);
  b += j * 2048;
  unsigned slot = 0;
#pragma unroll 1
  for (int t = 0; t < NSTEP; ++t) {
    BARRIER();
    const unsigned la = lbase + slot + j * (NU * 1024);
    const unsigned lb = lbase + slot + NU * 4096 + j * 2048;
    unsigned keep;
#define AG_A(off) "global_load_lds_dwordx4 %1, %2 offset:" #off ASC "\n\t"
#define AG_HEAD "s_mov_b32 %0, m0\n\ts_mov_b32 m0, %4\n\ts_nop 0\n\t"
#define AG_TAIL "s_mov_b32 m0, %5\n\ts_nop 0\n\t" \
                "global_load_lds_dwordx4 %1, %3" BSC "\n\t" \
                "global_load_lds_dwordx4 %1, %3 offset:1024" BSC "\n\t" \
                "s_mov_b32 m0, %0\n\ts_waitcnt vmcnt(%6)"
#define AG_OPS : "=&s"(keep) : "v"(voff), "s"(a), "s"(b), "s"(la), "s"(lb), "i"((D - 1) * NLD), \
                 "v"(voff + 4096), "s"(la + 4096) : "memory"
    if constexpr (NU == 1) asm volatile(AG_HEAD AG_A(0) AG_TAIL AG_OPS);
    if constexpr (NU == 2) asm volatile(AG_HEAD AG_A(0) AG_A(1024) AG_TAIL AG_OPS);
    if constexpr (NU == 3) asm volatile(AG_HEAD AG_A(0) AG_A(1024) AG_A(2048) AG_TAIL AG_OPS);
    if constexpr (NU == 4) asm volatile(AG_HEAD AG_A(0) AG_A(1024) AG_A(2048) AG_A(3072) AG_TAIL AG_OPS);
    if constexpr (NU == 5)
      asm volatile(AG_HEAD AG_A(0) AG_A(1024) AG_A(2048) AG_A(3072)
                   "s_mov_b32 m0, %8\n\ts_nop 0\n\tglobal_load_lds_dwordx4 %7, %2" ASC "\n\t" AG_TAIL AG_OPS);
    a += astep;
    b += 16384;
    slot += SLOT;
    if (slot == NS * SLOT) slot = 0;
  }
  static_for<D>([&](auto tc) {
    constexpr int t = decltype(tc)::value;
    BARRIER();
    wait_vm<(D - 2 - t > 0 ? D - 2 - t : 0) * NLD>();
  });
  BARRIER();
}

template <int MH>
__device__ __forceinline__ void tile_mfma(LDS_AS char* smem, const Tile& T, unsigned call, int unit, int m0, int lane,
                                          unsigned long long* dbg) {
  const int nu = T.nu;
  const int D = ring_slots(nu) - 1;
  LDS_AS unsigned* bad = (LDS_AS unsigned*)(smem + LDS_RING);
  f32x4 acc[2][4];
  for (int m = 0; m < 2; ++m)
    for (int n = 0; n < 4; ++n) acc[m][n] = f32x4{0.f, 0.f, 0.f, 0.f};
  f16x8 fa[2][2][MH], fb[2][4][MH];
  LDS_AS char* pa = smem + unit * 4096 + m0 * 1024 + lane * 16;
  LDS_AS char* pb = smem + nu * 4096 + m0 * 4096 + lane * 16;
  const unsigned flip = m0 * 2048;
  const unsigned sstep = nu * 4096 + 8192, send = sstep * (D + 1);
  unsigned slot = 0;
  auto rd = [&](auto sc) {
    constexpr int s = decltype(sc)::value;
    for (int u = 0; u < MH; ++u) {
      for (int m = 0; m < 2; ++m) fa[s][m][u] = *(LDS_AS f16x8*)(pa + slot + ((m * 2048) ^ flip) + u * 1024);
      for (int n = 0; n < 4; ++n) fb[s][n][u] = *(LDS_AS f16x8*)(pb + slot + u * 4096 + n * 1024);
    }
    slot += sstep;
    if (slot == send) slot = 0;
  };
  auto mm = [&](auto sc) {
    constexpr int s = decltype(sc)::value;
    for (int u = 0; u < MH; ++u)
      for (int m = 0; m < 2; ++m)
        for (int n = 0; n < 4; ++n)
          acc[m][n] = __builtin_amdgcn_mfma_f32_16x16x32_f16(fa[s][m][u], fb[s][n][u], acc[m][n], 0, 0, 0);
  };
  using I0 = std::integral_constant<int, 0>;
  using I1 = std::integral_constant<int, 1>;
  for (int t = 0; t <= D; ++t) BARRIER();
  rd(I0{});
  for (int s = 1; s < NSTEP - 1; s += 2) {
    BARRIER();
    rd(I1{});
    __builtin_amdgcn_sched_barrier(0);
    mm(I0{});
    __builtin_amdgcn_sched_barrier(0);
    BARRIER();
    rd(I0{});
    __builtin_amdgcn_sched_barrier(0);
    mm(I1{});
    __builtin_amdgcn_sched_barrier(0);
  }
  BARRIER();
  rd(I1{});
  const unsigned isbad = *bad;
  __builtin_amdgcn_sched_barrier(0);
  mm(I0{});
  __builtin_amdgcn_sched_barrier(0);
  BARRIER();
  mm(I1{});
  if constexpr (MH == 1) {
    volatile LDS_AS f32x4* swap = (volatile LDS_AS f32x4*)(smem + unit * 8192) + lane;
    for (int n = 0; n < 4; ++n) swap[m0 * 256 + n * 64] = acc[1][n];
    asm volatile("s_waitcnt lgkmcnt(0)" ::: "memory");
    if (first_lane(lane)) __hip_atomic_fetch_add(bad + 1 + unit, 1u, __ATOMIC_RELAXED, __HIP_MEMORY_SCOPE_WORKGROUP);
    while (*(volatile LDS_AS unsigned*)(bad + 1 + unit) < 2) {}
    asm volatile("" ::: "memory");
    for (int n = 0; n < 4; ++n) acc[0][n] += swap[(m0 ^ 1) * 256 + n * 64];
  }
  char* out = uniform(T.out + (size_t)unit * (T.staged ? 2 * 8192 : 32 * SHARD * 4));
  const unsigned sj = T.staged ? 1024 : SHARD * 4, sm = T.staged ? 4096 : 16 * SHARD * 4;
  const unsigned vo = m0 * sm + (T.staged ? lane * 16 : (unsigned)(4 * (lane >> 4)) * (SHARD * 4) + (lane & 15) * 16);
  for (int m = 0; m < MH; ++m) {
    for (int jj = 0; jj < 4; ++jj) {
      u32x4 o;
      for (int n = 0; n < 4; ++n)
        o[n] = isbad | __builtin_bit_cast(unsigned, gelu(acc[m][n][jj]));
      asm volatile("global_store_dwordx4 %1, %0, %2 sc0 sc1\n\ts_nop 1"
                   :: "v"(o), "v"(vo + m * sm + jj * sj), "s"(out) : "memory");
    }
  }
  if (T.oflag != nullptr) {
    wait_vm<0>();
    if (first_lane(lane) &&
        (MH == 2 || __hip_atomic_fetch_add(bad + 1 + unit, 1u, __ATOMIC_RELAXED, __HIP_MEMORY_SCOPE_WORKGROUP) == 3))
      flag_store(T.oflag + (T.staged ? flag_at(0, T.piece + 2 * unit) : unit * 256), call);
  }
  if (unit + m0 == 0) STAMP(dbg, 5);
}

__device__ __forceinline__ void gemm_tile(LDS_AS char* smem, char* ws, const Tile& T, unsigned call,
                                          int wave, int lane, unsigned long long* dbg) {
  const int nu = T.nu;
  const int D = ring_slots(nu) - 1;
  LDS_AS unsigned* bad = (LDS_AS unsigned*)(smem + LDS_RING);

  if (wave == 9) {
    const bool lo = lane < 48;
    const int g = lo ? (lane & 15) + (lane >> 4) * 64 : lane - 48;
    const char* fp = ws + OFF_DONE + (call & 1) * 131072 + (lo ? T.aflag + (lane >> 4) * 16 : T.bflag) + (lane & 15) * 128;
    const int per = lo ? T.au : 2;
    char* tmo = ws + OFF_TMO;
    int first = g < 32 * per ? g : 4096;
    int need = per;
    int ahead = 4096, ask = 0;
    bool dead = false;
    volatile LDS_AS u32x4* news = (volatile LDS_AS u32x4*)(smem + LDS_RING + 64) + lane;
    *news = u32x4{call - 1, call - 1, call - 1, call - 1};
    STAMP(dbg, 0);
    for (int t = 0; t < NSTEP; ++t) {
      if (NEAR && !__all(2 * first >= min(need + NEAR * per, NSTEP * per))) {
        first = max(first, first_unset(*news, call, g));
        if (t >= ask && __all(2 * first >= need)) {
          unsigned keep;
          asm volatile("s_mov_b32 %0, m0\n\ts_mov_b32 m0, %2\n\ts_nop 0\n\t"
                       "global_load_lds_dwordx4 %1, off sc0 sc1\n\ts_mov_b32 m0, %0"
                       : "=&s"(keep) : "v"(fp), "s"((unsigned)(uintptr_t)(smem + LDS_RING + 64)) : "memory");
          ask = t + EVERY;
        }
      }
      if (!dead && !__all(2 * first >= need)) {
        const unsigned long long t0 = __builtin_amdgcn_s_memrealtime();
        for (;;) {
          look(first, 2 * first < need + ahead, fp, call, g);
          ahead = AHEAD;
          if (__all(2 * first >= need)) break;
          if (expired(tmo, call, t0, lane)) { dead = true; break; }
          if (t == 0) __builtin_amdgcn_s_sleep(4 * POLL_SLEEP); else __builtin_amdgcn_s_sleep(POLL_SLEEP);
        }
      }
      need += per;
      BARRIER();
    }
    if (first_lane(lane)) *(LDS_AS u32x4*)bad = u32x4{dead ? 0x7fc00000u : 0u, 0, 0, 0};
    for (int t = 0; t <= D; ++t) BARRIER();
    wait_vm<0>();
    STAMP(dbg, 4);
  } else if (wave >= 5) {
    const char* data = ws + OFF_DATA + (size_t)(call & 1) * PARITY_B;
    const unsigned lbase = (unsigned)(uintptr_t)smem;
    const char* a = data + T.a;
    const char* b = data + T.b;
    const int j = wave - 5;
    if (nu == 4) tile_loader<4>(lbase, a, T.au * 4096, b, j, lane);
    else if (nu == 2) tile_loader<2>(lbase, a, T.au * 4096, b, j, lane);
    else if (nu == 1) tile_loader<1>(lbase, a, T.au * 4096, b, j, lane);
    else if (nu == 3) tile_loader<3>(lbase, a, T.au * 4096, b, j, lane);
    else tile_loader<5>(lbase, a, T.au * 4096, b, j, lane);
  } else if (wave >= (nu <= SMALL ? 2 * nu : nu)) {
    for (int t = 0; t < NSTEP + D + 1; ++t) BARRIER();
  } else if (nu <= SMALL) {
    tile_mfma<1>(smem, T, call, wave >> 1, wave & 1, lane, dbg);
  } else {
    tile_mfma<2>(smem, T, call, wave, 0, lane, dbg);
  }
}

__device__ __forceinline__ Tile make_tile(const Params& P, int src, int u0, int nu, bool foreign, int xcd,
                                          unsigned call) {
  const int grp = P.rank >> 2, pos = P.rank & 3;
  const int c = xcd >> 1, ch = xcd & 1, cr = (4 - c) & 3;
  Tile T;
  int row0, R;
  bool direct = false;
  if (src < 4) {
    const int ub = src < 3 ? 256 * src : 512 + 64 * U2;
    T.au = src < 2 ? 4 : src == 2 ? U2 : UX;
    T.a = D_OWNX + ub * 4096;
    row0 = P.rank * SHARD + (ub >> 1);
    R = ub >> 6;
  } else if (src < 13) {
    const int i = (src - 4) / 3, b = src - 4 - 3 * i;
    T.au = b < 2 ? 4 : U2;
    T.a = D_INTRA + (3 * i + b) * MIB;
    row0 = (grp * 4 + ((pos + i + 1) & 3)) * SHARD + b * 128;
    R = b < 2 ? 8 + U2 + 12 * b + 4 * i : U2 * i;
    direct = b == 2;
  } else {
    const int k = src - 13;
    T.au = UX;
    T.a = D_CROSS + (3 * k + 1) * MIB;
    row0 = ((grp ^ 1) * 4 + ((pos + k) & 3)) * SHARD + ROWX;
    R = 32 + U2 + ((k - c - 1) & 3) * UX;
  }
  T.nu = nu;
  T.aflag = (src == 3 ? 3 : T.a >> 20) * 2048;
  T.a += u0 * 4096;
  row0 += u0 * 32;
  R += u0;
  T.staged = foreign && !direct;
  T.b = foreign ? D_CROSS + 3 * c * MIB : D_OWNW + c * MIB;
  T.bflag = (T.b >> 20) * 2048 + ch * 16;
  T.piece = 2 * R + ch;
  T.b += ch * 8192;
  if (!foreign) {
    T.out = (char*)P.out[P.rank & 7] + (size_t)row0 * (SHARD * 4) + xcd * 256;
    T.oflag = nullptr;
  } else {
    if (direct) {
      const int drank = ((grp ^ 1) * 4 + ((pos + c) & 3)) & 7;
      T.out = (char*)P.out[drank] + (size_t)row0 * (SHARD * 4) + cr * 512 + ch * 256;
      T.oflag = P.ws[drank] + OFF_OFLAG + (size_t)(cr * 64 + NWV + 2 * R + ch) * 128;
    } else {
      T.out = P.self + OFF_DATA + (size_t)(call & 1) * PARITY_B + D_STG + c * MIB + (2 * R + ch) * 8192;
      T.oflag = P.self + OFF_DONE + (call & 1) * 131072 + (D_STG / MIB + c) * 2048;
    }
  }
  return T;
}

__global__ void __launch_bounds__(NTHREADS, 1) ag_gemm_kernel(Params P) {
  __shared__ __attribute__((aligned(16))) char smem_raw[LDS_BYTES];
  LDS_AS char* smem = (LDS_AS char*)smem_raw;
  const int lane = threadIdx.x & 63;
  const int wave = __builtin_amdgcn_readfirstlane(threadIdx.x >> 6);
  const int bx = blockIdx.x, xcd = bx & 7, j = bx >> 3;
  const int grp = P.rank >> 2, pos = P.rank & 3;
  char* ws = P.self;
  unsigned* ctr = (unsigned*)(ws + OFF_CTR + (size_t)bx * 128);
  unsigned long long* dbg = (unsigned long long*)(ws + OFF_DBG + (size_t)bx * 512);
  if (wave == 9) STAMP(dbg, 0);

  const int sub = (xcd + j) & 7;
  if (j < JP && wave < 9) {
    const char* tick = ws + OFF_TICK + j * 256;
    if (wave >= PW) return;
    if (j < 3)
      send_stream<false>(P, (grp * 4 + ((pos + j + 1) & 3)) & 7, 0, D_INTRA / MIB + (2 - j) * 3, ctr, tick,
                         sub * PW + wave, lane, dbg + 16 * wave);
    else
      send_stream<true>(P, ((grp ^ 1) * 4 + ((pos + j - 3) & 3)) & 7, j - 3, D_CROSS / MIB + ((7 - j) & 3) * 3, ctr,
                        tick, sub * PW + wave, lane, dbg + 16 * wave);
    return;
  }
  const unsigned call = *(const CONST_AS unsigned*)(uintptr_t)ctr + 1u;
  asm volatile("" :: "s"(call));
  if (j < JP) {
    const int c = j - 3;
    BARRIER();
    if (j >= 3 && sub == 0) {
      const char* fp = ws + OFF_OFLAG + (size_t)(c * 64 + (lane < NWV + NDIR ? lane : 0)) * 128;
      char* tmo = ws + OFF_TMO;
      const unsigned long long t0 = __builtin_amdgcn_s_memrealtime();
      while (__builtin_amdgcn_s_memrealtime() - t0 < 4500) __builtin_amdgcn_s_sleep(32);
      for (;;) {
        const unsigned f = __hip_atomic_load((GLOBAL_AS unsigned*)fp, __ATOMIC_RELAXED, __HIP_MEMORY_SCOPE_SYSTEM);
        if (__all((int)(f - call) >= 0)) break;
        if (expired(tmo, call, t0, lane)) {
          if (first_lane(lane)) flag_store(P.out[P.rank & 7] + (size_t)P.rank * SHARD * SHARD, 0x7fc00000u);
          break;
        }
        __builtin_amdgcn_s_sleep(2);
      }
      STAMP(dbg, 62);
    }
  } else {
    const int r = j - JP;
    if (r >= 9 && wave < 8) {
      const int q = (r - 9) * 8 + xcd, t = q >> 1;
      unsigned own = 512 + (wave >> 1) * 128 + 2 * t + (wave & 1);
      if (!(q & 1)) {
        const int i = (t & 1) * 8 + wave, s = t >> 1;
        own = i < 8 ? (i >> 2) * 128 + 4 * s + (i & 3) : i < 8 + U2 ? 256 + U2 * s + i - 8
                                                                     : 256 + 32 * U2 + UX * s + i - 8 - U2;
      }
      const unsigned wait = (r < 9 + 2 * NH ? CONV_D0 : (r - 9 - 2 * NH) * CONV_DL) + (own >= 128 && own < 512 ? CONV_NU : 0);
      const unsigned long long t0 = __builtin_amdgcn_s_memrealtime();
      while (__builtin_amdgcn_s_memrealtime() - t0 < wait) __builtin_amdgcn_s_sleep(2);
      convert_chunk(P, own, call, (LDS_AS unsigned short*)(smem + wave * 1024), lane);
      if (wave == 0) STAMP(dbg, 40);
    }
    if (r >= 20) {
      const int w = wave * 5 + r - 20, link = w % 7, k = w / 7;
      if (k < UNP)
        unpack_stream(ws, link, call, (LDS_AS unsigned*)(smem + wave * 1024), k * 8 + ((xcd + link) & 7), lane,
                      wave == 0 ? dbg + 56 : wave == 5 ? dbg + 48 : dbg + 64 * 512);
    } else if (r >= 13 && wave == 9) {
      const unsigned long long t0 = __builtin_amdgcn_s_memrealtime();
      while (__builtin_amdgcn_s_memrealtime() - t0 < LATE) __builtin_amdgcn_s_sleep(16);
    }
    const int cx = 13 + (((xcd >> 1) + 1 + r % 3) & 3), i3 = 3 * (r % 3);
#define TD(src, u0, nu, foreign) ((src) | (u0) << 5 | (nu) << 7 | (foreign) << 10)
    int d0, d1 = 0, d2 = 0;
    if (r < 3) { d0 = TD(r, 0, r < 2 ? 4 : 3, 1); d1 = TD(5 + i3, 0, 2, 1); d2 = TD(6 + i3, 2, 1, 0); }
    else if (r < 6) { d0 = TD(4 + i3, 0, 4, 1); d1 = TD(cx, 3, 2, 1); }
    else if (r < 9) { d0 = TD(4 + i3, 0, 4, 0); d1 = TD(cx, 0, 3, 1); }
    else if (r < 12) { d0 = TD(r - 9, 0, r < 11 ? 4 : 3, 0); d1 = TD(5 + i3, 2, 2, 1); d2 = TD(6 + i3, 0, 2, 0); }
    else if (r == 12) { d0 = TD(3, 0, 5, 0); d1 = TD(6, 2, 1, 1); }
    else if (r < 16) d0 = TD(5 + i3, 0, 4, 0);
    else if (r < 20) d0 = TD(r - 3, 0, 5, 0);
    else if (r < 23) d0 = TD(6 + i3, 0, 2, 1);
    else d0 = TD(6 + 3 * (r - 22), 2, 1, 1);
#pragma unroll 1
    for (int n = 0; d0 != 0; ++n, d0 = d1, d1 = d2, d2 = 0) {
      const Tile T = make_tile(P, d0 & 31, (d0 >> 5) & 3, (d0 >> 7) & 7, d0 >> 10, xcd, call);
      gemm_tile(smem, ws, T, call, wave, lane, dbg + 8 + 8 * n);
    }
  }
  if (wave == 9) {
    if (first_lane(lane)) __hip_atomic_store((GLOBAL_AS unsigned*)ctr, call, __ATOMIC_RELAXED, __HIP_MEMORY_SCOPE_AGENT);
    STAMP(dbg, 63);
  }
}

extern "C" void kernel_launch(int rank, int nranks, ncclComm_t comm,
                              hipStream_t stream,
                              void* const* d_in, const int* in_sizes, int n_in,
                              void* d_out, int out_size,
                              const int* in_dtypes, int out_dtype,
                              void* const* d_out_all,
                              void* const* d_ws_all, size_t ws_bytes) {
  (void)comm; (void)in_dtypes; (void)out_dtype; (void)d_out;
  if (nranks != NRANK || (unsigned)rank >= (unsigned)NRANK || n_in < 2 || ws_bytes < WS_NEED ||
      in_sizes[0] != SHARD * KDIM || in_sizes[1] != KDIM * SHARD || out_size != NRANK * SHARD * SHARD) {
    fprintf(stderr, "unexpected problem shape\n");
    return;
  }
  Params P;
  P.x = (const float*)d_in[0];
  P.w = (const float*)d_in[1];
  for (int p = 0; p < NRANK; ++p) {
    P.ws[p] = (char*)d_ws_all[p];
    P.out[p] = (float*)d_out_all[p];
  }
  P.self = (char*)d_ws_all[rank];
  P.rank = rank;
  P.pad = 0;
  ag_gemm_kernel<<<NWG, NTHREADS, 0, stream>>>(P);
  HIP_CHECK(hipGetLastError());
}
```
